```python
import numpy as np

kernel_name = "gpu_ag_gemm_m4096_k4096_n4096_e5m2e5m2_dqsilu"


ABSMAX_RTOL = 0.05

COMPUTE_DTYPE = "float32"
EPILOGUE = "dequant_silu"
IN_QUANT = None
OUT_QUANT = None
IN_FP8 = "e5m2e5m2"
IN_DTYPES = {"x": "e5m2", "w_mat": "e5m2",
             "scale_x": "float32", "scale_w": "float32"}
WORLD_SIZE = 8

def _snap_fp8(x, mant_bits, max_abs, min_normal):
    a = np.abs(x.astype(np.float64))
    _m, _e = np.frexp(a)
    q = np.where(a >= min_normal, np.ldexp(1.0, _e - 1 - mant_bits),
                 min_normal * 2.0 ** -mant_bits)
    snapped = np.minimum(np.round(a / q) * q, max_abs)
    return (np.sign(x) * snapped).astype(np.float32)

def setup_inputs():

    x = _snap_fp8(np.random.randn(4096, 4096).astype(np.float32), 2, 57344.0, 2.0 ** -14)
    w_mat = _snap_fp8(np.random.randn(4096, 4096).astype(np.float32), 2, 57344.0, 2.0 ** -14)
    scale_x = (0.02 * (1.0 + np.random.rand(1))).astype(np.float32)
    scale_w = (0.02 * (1.0 + np.random.rand(1))).astype(np.float32)
    return {"x": x, "w_mat": w_mat, "scale_x": scale_x, "scale_w": scale_w}

def reference(x, w_mat, scale_x, scale_w):
    acc = (x @ w_mat).astype(np.float32)
    y = (acc * (scale_x * scale_w)).astype(np.float32)
    return (y / (1.0 + np.exp(-np.clip(y, -60.0, 60.0)))).astype(np.float32)
```

```cpp
#include <hip/hip_runtime.h>
#include <rccl/rccl.h>
#include <cstdio>
#include <cstdint>

#define HIP_CHECK(expr) do { hipError_t _e = (expr); if (_e != hipSuccess) { \
    fprintf(stderr, "HIP error %s at %s:%d\n", hipGetErrorString(_e), __FILE__, __LINE__); \
    return; } } while (0)

namespace {

typedef unsigned int u32x4 __attribute__((ext_vector_type(4)));
typedef int v2i __attribute__((ext_vector_type(2)));
typedef int v8i __attribute__((ext_vector_type(8)));
typedef float f32x4 __attribute__((ext_vector_type(4)));
typedef float f32x16 __attribute__((ext_vector_type(16)));

constexpr int NR = 8;
constexpr int KD = 4096;
constexpr int MSH = 512;
constexpr int NSH = 512;
constexpr unsigned SHARD_BYTES = MSH * KD;
constexpr size_t XBUF_BYTES = 2ull * NR * SHARD_BYTES;
constexpr size_t FLAG_OFF = XBUF_BYTES;
constexpr size_t CTR_OFF = FLAG_OFF + NR * 32 * 128;
constexpr size_t WS_NEED = CTR_OFF + 256 * 128;
constexpr unsigned long long DEADLINE_TICKS = 50000000ull;

constexpr int NWG = 256;
constexpr int NTHREADS = 576;
constexpr int WIN = 5;
constexpr int LDS_SYNC_OFF = 8 * 16384;
constexpr int LDS_BYTES = LDS_SYNC_OFF + 128;

struct Params {
    const uint8_t* x;
    const uint8_t* w;
    const float* sx;
    const float* sw;
    float* out;
    char* ws[NR];
    int rank;
};

__device__ __forceinline__ __amdgpu_buffer_rsrc_t mkrsrc(const void* p, unsigned bytes) {
    unsigned long long a = (unsigned long long)p;
    unsigned lo = __builtin_amdgcn_readfirstlane((unsigned)a);
    unsigned hi = __builtin_amdgcn_readfirstlane((unsigned)(a >> 32));
    return __builtin_amdgcn_make_buffer_rsrc((void*)(((unsigned long long)hi << 32) | lo), 0, bytes, 0x00020000);
}
template <int N> __device__ __forceinline__ void wait_vm() { asm volatile("s_waitcnt vmcnt(%0)" :: "i"(N) : "memory"); }
__device__ __forceinline__ void wait_lgkm0() { asm volatile("s_waitcnt lgkmcnt(0)" ::: "memory"); }

__device__ __host__ constexpr int flag_need(int j) { return j < 2 ? 16 : j < 4 ? 32 : j < 6 ? 48 : j < 7 ? 56 : 64; }
__device__ __host__ constexpr bool flag_point(int n) { return n == 16 || n == 32 || n == 48 || n == 56; }
__device__ __host__ constexpr int younger_than(int n) {
    int c = WIN;
    for (int m = n - WIN; m <= n; ++m) c += (m + WIN < 64) ? 1 : 0;
    return c;
}
typedef __attribute__((address_space(3))) unsigned lds_u32;

#define STAMP(i) do { } while (0)

__global__ void __launch_bounds__(NTHREADS) ag_gemm_kernel(Params p) {
    __shared__ __attribute__((aligned(16))) char lds[LDS_BYTES];
    const int b = blockIdx.x;
    const int lane = threadIdx.x & 63;
    const int wave = __builtin_amdgcn_readfirstlane(threadIdx.x >> 6);
    const int q = b & 7;
    const int idx = b >> 3;
    const int peer = (p.rank + q) & 7;
    const unsigned long long t_start = __builtin_amdgcn_s_memrealtime();
    lds_u32* sync_words = (lds_u32*)(lds + LDS_SYNC_OFF);
    STAMP(0);

    if (wave == 0) {
        const int g = idx >> 4, w = (idx >> 1) & 7, u = idx & 1;
        __amdgpu_buffer_rsrc_t rs = mkrsrc(p.x, SHARD_BYTES);
        const unsigned so = (unsigned)(g * 256 + (lane & 31)) * KD + w * 512 + u * 256 + (lane >> 5) * 32;
        unsigned* ctr = (unsigned*)(p.ws[p.rank] + CTR_OFF) + b * 32;
        const unsigned prev = __hip_atomic_load(ctr, __ATOMIC_RELAXED, __HIP_MEMORY_SCOPE_AGENT);
        u32x4 v[WIN];
#pragma unroll
        for (int n = 0; n < WIN; ++n)
            v[n] = __builtin_amdgcn_raw_buffer_load_b128(rs, so + (n >> 3) * 32 * KD + ((n >> 1) & 3) * 64 + (n & 1) * 16, 0, 0);
        const unsigned call = __builtin_amdgcn_readfirstlane(prev) + 1u;
        if (lane < 9) __hip_atomic_store(sync_words + lane, lane == 8 ? call : 0u, __ATOMIC_RELAXED, __HIP_MEMORY_SCOPE_WORKGROUP);
        wait_lgkm0();
        __builtin_amdgcn_s_barrier();
        STAMP(1);
        __amdgpu_buffer_rsrc_t rd = mkrsrc(p.ws[peer] + (size_t)((call & 1u) * NR + p.rank) * SHARD_BYTES, SHARD_BYTES);
        unsigned* rflag = (unsigned*)(p.ws[peer] + FLAG_OFF) + (p.rank * 32 + idx) * 32;
        const unsigned dofs = (unsigned)((g * 64 + w) * 16 + u * 8) * 1024u + lane * 16u;
        const unsigned base = call * 128u;
#pragma unroll
        for (int n = 0; n < 64; ++n) {
            __builtin_amdgcn_raw_buffer_store_b128(v[n % WIN], rd, dofs + (n >> 3) * (8 * 16 * 1024) + (n & 7) * 1024, 0, 17);
            const int m = n + WIN;
            if (m < 64)
                v[n % WIN] = __builtin_amdgcn_raw_buffer_load_b128(rs, so + (m >> 3) * 32 * KD + ((m >> 1) & 3) * 64 + (m & 1) * 16, 0, 0);
            if (flag_point(n - WIN + 1)) {
                if (n - WIN + 1 == 16) wait_vm<younger_than(16 + WIN - 1)>();
                else if (n - WIN + 1 == 32) wait_vm<younger_than(32 + WIN - 1)>();
                else if (n - WIN + 1 == 48) wait_vm<younger_than(48 + WIN - 1)>();
                else wait_vm<younger_than(56 + WIN - 1)>();
                if (lane == 0) __hip_atomic_store(rflag, base + (unsigned)(n - WIN + 1), __ATOMIC_RELAXED, __HIP_MEMORY_SCOPE_SYSTEM);
            }
        }
        wait_vm<0>();
        if (lane == 0) {
            __hip_atomic_store(rflag, base + 64u, __ATOMIC_RELAXED, __HIP_MEMORY_SCOPE_SYSTEM);
            __hip_atomic_store(ctr, call, __ATOMIC_RELAXED, __HIP_MEMORY_SCOPE_AGENT);
        }
        STAMP(2);
        return;
    }

    const int w = wave - 1;
    const int g = idx >> 4, ct = idx & 15;
    char* area = lds + w * 16384;
    {
        const uint8_t* gsrc = p.w + (size_t)(w * 512 + (lane >> 1)) * NSH + ct * 32 + (lane & 1) * 16;
#pragma unroll
        for (int t = 0; t < 16; ++t)
            __builtin_amdgcn_global_load_lds((const __attribute__((address_space(1))) void*)(gsrc + (size_t)t * 32 * NSH),
                                             (__attribute__((address_space(3))) void*)(area + t * 1024), 16, 0, 0);
    }
    const float scale = p.sx[0] * p.sw[0];
    __builtin_amdgcn_s_barrier();
    asm volatile("" ::: "memory");
    const unsigned call = __builtin_amdgcn_readfirstlane(__hip_atomic_load(sync_words + 8, __ATOMIC_RELAXED, __HIP_MEMORY_SCOPE_WORKGROUP));
    const unsigned base = call * 128u;
    wait_vm<0>();
    v8i wf[8];
    {
        const int grp = lane >> 4, i = lane & 15;
        const unsigned a0 = (unsigned)(((grp >> 1) * 32 + (i >> 1)) * 32 + (grp & 1) * 16 + (i & 1) * 8);
#pragma unroll
        for (int s = 0; s < 8; ++s)
#pragma unroll
            for (int j = 0; j < 4; ++j) {
                v2i r = __builtin_amdgcn_ds_read_tr8_b64_v2i32(
                    (__attribute__((address_space(3))) v2i*)(area + a0 + (s * 64 + j * 8) * 32));
                wf[s][2 * j] = r[0];
                wf[s][2 * j + 1] = r[1];
            }
    }
    STAMP(1);
    __amdgpu_buffer_rsrc_t rx = mkrsrc(p.ws[p.rank] + (size_t)((call & 1u) * NR + peer) * SHARD_BYTES, SHARD_BYTES);
    const unsigned* flag = (const unsigned*)(p.ws[p.rank] + FLAG_OFF) + (peer * 32 + g * 16 + w * 2 + (lane & 1)) * 32;
    const unsigned xo = (unsigned)((g * 64 + w) * 16) * 1024u + lane * 16u;
    const int m_w = lane & 31, h_w = lane >> 5;
    const int m_r = w * 4 + ((lane & 31) >> 3), c_r = lane & 7;
    const unsigned rd_off = (unsigned)(m_r * 128 + ((c_r ^ (m_r & 7)) << 4));
    float* outp = p.out + (size_t)(peer * MSH + g * 256 + m_r) * NSH + ct * 32 + c_r * 4;
    bool bad = false;

#pragma unroll 1
    for (int j = 0; j < 8; ++j) {
        const unsigned want = base + (unsigned)flag_need(j);
        if (!bad) {
            for (unsigned spin = 1;; ++spin) {
                const unsigned f = __hip_atomic_load(flag, __ATOMIC_RELAXED, __HIP_MEMORY_SCOPE_SYSTEM);
                if (__all((int)(f - want) >= 0)) break;
                if ((spin & 63u) == 0u && __builtin_amdgcn_s_memrealtime() - t_start > DEADLINE_TICKS) { bad = true; break; }
                __builtin_amdgcn_s_sleep(2);
            }
        }
        asm volatile("" ::: "memory");
        STAMP(2 + 3 * j);
        f32x16 acc = {};
        if (!bad) {
            u32x4 xf[16];
#pragma unroll
            for (int n = 0; n < 16; ++n)
                xf[n] = __builtin_amdgcn_raw_buffer_load_b128(rx, xo + j * (8 * 16 * 1024) + n * 1024, 0, 16);
            __builtin_amdgcn_sched_barrier(0);
#pragma unroll
            for (int s = 0; s < 8; ++s) {
                v8i xa;
                xa[0] = xf[2 * s][0]; xa[1] = xf[2 * s][1]; xa[2] = xf[2 * s][2]; xa[3] = xf[2 * s][3];
                xa[4] = xf[2 * s + 1][0]; xa[5] = xf[2 * s + 1][1]; xa[6] = xf[2 * s + 1][2]; xa[7] = xf[2 * s + 1][3];
                acc = __builtin_amdgcn_mfma_scale_f32_32x32x64_f8f6f4(wf[s], xa, acc, 1, 1, 0, 0, 0, 0);
            }
        } else {
#pragma unroll
            for (int i = 0; i < 16; ++i) acc[i] = __builtin_nanf("");
        }
        STAMP(3 + 3 * j);
        char* slot = area + (j & 1) * 4096;
#pragma unroll
        for (int t = 0; t < 4; ++t) {
            f32x4 vv = {acc[4 * t], acc[4 * t + 1], acc[4 * t + 2], acc[4 * t + 3]};
            *(f32x4*)(slot + m_w * 128 + (((2 * t + h_w) ^ (m_w & 7)) << 4)) = vv;
        }
        wait_lgkm0();
        if (lane == 0) __hip_atomic_fetch_add(sync_words + j, 1u, __ATOMIC_RELAXED, __HIP_MEMORY_SCOPE_WORKGROUP);
        for (unsigned spin = 1; __builtin_amdgcn_readfirstlane(__hip_atomic_load(sync_words + j, __ATOMIC_RELAXED, __HIP_MEMORY_SCOPE_WORKGROUP)) < 8u; ++spin) {
            if ((spin & 1023u) == 0u && __builtin_amdgcn_s_memrealtime() - t_start > 2 * DEADLINE_TICKS) { bad = true; break; }
        }
        asm volatile("" ::: "memory");
        if (lane < 32) {
            f32x4 sum = *(const f32x4*)(lds + (j & 1) * 4096 + rd_off);
#pragma unroll
            for (int v = 1; v < 8; ++v) sum += *(const f32x4*)(lds + v * 16384 + (j & 1) * 4096 + rd_off);
            f32x4 o;
#pragma unroll
            for (int i = 0; i < 4; ++i) {
                const float y = sum[i] * scale;
                const float yc = fminf(fmaxf(y, -60.0f), 60.0f);
                o[i] = y * __builtin_amdgcn_rcpf(1.0f + __expf(-yc));
            }
            if (bad) o = f32x4{__builtin_nanf(""), __builtin_nanf(""), __builtin_nanf(""), __builtin_nanf("")};
            *(f32x4*)(outp + (size_t)j * 32 * NSH) = o;
        }
        STAMP(4 + 3 * j);
    }
}

}

extern "C" void kernel_launch(int rank, int nranks, ncclComm_t comm,
                              hipStream_t stream,
                              void* const* d_in, const int* in_sizes, int n_in,
                              void* d_out, int out_size,
                              const int* in_dtypes, int out_dtype,
                              void* const* d_out_all,
                              void* const* d_ws_all, size_t ws_bytes) {
    (void)comm; (void)in_dtypes; (void)out_dtype; (void)d_out_all;
    if (nranks != NR || (unsigned)rank >= (unsigned)NR || n_in < 4 || ws_bytes < WS_NEED ||
        in_sizes[0] != MSH * KD || in_sizes[1] != KD * NSH || out_size != NR * MSH * NSH) {
        fprintf(stderr, "kernel_launch: unexpected problem shape\n");
        return;
    }
    Params p;
    p.x = (const uint8_t*)d_in[0];
    p.w = (const uint8_t*)d_in[1];
    p.sx = (const float*)d_in[2];
    p.sw = (const float*)d_in[3];
    p.out = (float*)d_out;
    for (int r = 0; r < NR; ++r) p.ws[r] = (char*)d_ws_all[r];
    p.rank = rank;
    hipLaunchKernelGGL(ag_gemm_kernel, dim3(NWG), dim3(NTHREADS), 0, stream, p);
    HIP_CHECK(hipGetLastError());
}
```

```cpp
#include <hip/hip_runtime.h>
#include <rccl/rccl.h>
#include <cstdio>
#include <cstdint>

#define HIP_CHECK(expr) do { hipError_t _e = (expr); if (_e != hipSuccess) { \
    fprintf(stderr, "HIP error %s at %s:%d\n", hipGetErrorString(_e), __FILE__, __LINE__); \
    return; } } while (0)

namespace {

typedef unsigned int u32x4 __attribute__((ext_vector_type(4)));
typedef int v2i __attribute__((ext_vector_type(2)));
typedef int v8i __attribute__((ext_vector_type(8)));
typedef float f32x4 __attribute__((ext_vector_type(4)));
typedef float f32x16 __attribute__((ext_vector_type(16)));
typedef __attribute__((address_space(3))) unsigned lds_u32;
typedef __attribute__((address_space(3))) void lds_void;
typedef const __attribute__((address_space(1))) void glb_void;

constexpr int NR = 8;
constexpr int KD = 4096;
constexpr int MSH = 512;
constexpr int NSH = 512;
constexpr unsigned SHARD_BYTES = MSH * KD;
constexpr size_t XBUF_BYTES = 2ull * NR * SHARD_BYTES;
constexpr size_t FLAG_OFF = XBUF_BYTES;
constexpr size_t CTR_OFF = FLAG_OFF + NR * 16 * 128;
constexpr size_t WS_NEED = CTR_OFF + 256 * 128;
constexpr unsigned long long DEADLINE_TICKS = 50000000ull;

constexpr int NWG = 256;
constexpr int NTHREADS = 512;
#define AG_WIN 4
#define AG_OUT_AUX 16
#define AG_POLL_SCOPE __HIP_MEMORY_SCOPE_SYSTEM
#define AG_W_DELAY 32
#define AG_TAIL_WIN AG_WIN
constexpr int WIN = AG_WIN;
constexpr int LDS_SYNC_OFF = 8 * 16384;
constexpr int LDS_BYTES = LDS_SYNC_OFF + 128;

struct Params {
    const uint8_t* x;
    const uint8_t* w;
    const float* sx;
    const float* sw;
    float* out;
    char* ws[NR];
    int rank;
};

__device__ __forceinline__ __amdgpu_buffer_rsrc_t mkrsrc(const void* p, unsigned bytes) {
    unsigned long long a = (unsigned long long)p;
    unsigned lo = __builtin_amdgcn_readfirstlane((unsigned)a);
    unsigned hi = __builtin_amdgcn_readfirstlane((unsigned)(a >> 32));
    return __builtin_amdgcn_make_buffer_rsrc((void*)(((unsigned long long)hi << 32) | lo), 0, bytes, 0x00020000);
}
template <int N> __device__ __forceinline__ void wait_vm() { asm volatile("s_waitcnt vmcnt(%0)" :: "i"(N) : "memory"); }
__device__ __forceinline__ void wait_lgkm0() { asm volatile("s_waitcnt lgkmcnt(0)" ::: "memory"); }
__device__ __forceinline__ unsigned lds_load(lds_u32* p) {
    return __builtin_amdgcn_readfirstlane(__hip_atomic_load(p, __ATOMIC_RELAXED, __HIP_MEMORY_SCOPE_WORKGROUP));
}

__device__ __forceinline__ void lds_add1_asm(lds_u32* p) {
    asm volatile("ds_add_u32 %0, %1" :: "v"((unsigned)(uintptr_t)p), "v"(1u) : "memory");
}
__device__ __forceinline__ void lds_store_asm(lds_u32* p, unsigned v) {
    asm volatile("ds_write_b32 %0, %1" :: "v"((unsigned)(uintptr_t)p), "v"(v) : "memory");
}

__device__ __forceinline__ unsigned flag_need(int j) { return j < 2 ? 32u : j < 4 ? 64u : j < 6 ? 96u : j < 7 ? 112u : 128u; }
__device__ __forceinline__ bool flag_point(int n) { return n == 32 || n == 64 || n == 96 || n == 112; }

#define STAMP(i) do { } while (0)
#define STAMP_FLUSH() do { } while (0)

__global__ void __launch_bounds__(NTHREADS, 2) ag_gemm_kernel(Params p) {
    __shared__ __attribute__((aligned(16))) char lds[LDS_BYTES];
    const int b = blockIdx.x;
    const int lane = threadIdx.x & 63;
    const int wave = __builtin_amdgcn_readfirstlane(threadIdx.x >> 6);
    const int q = b & 7;
    const int idx = b >> 3;
#define AG_MAP 0
    const int peer = AG_MAP == 0 ? (p.rank + q) & 7 : AG_MAP == 1 ? (p.rank - q) & 7 : AG_MAP == 2 ? (p.rank ^ q) : q;
    const int g = (idx >> 3) & 1;
    const unsigned long long t_start = __builtin_amdgcn_s_memrealtime();
    lds_u32* sync_words = (lds_u32*)(lds + LDS_SYNC_OFF);
    unsigned* ctr = (unsigned*)(p.ws[p.rank] + CTR_OFF) + b * 32;
    STAMP(0);

    if (idx < 16) {
        const int w = idx & 7;
        if (wave == 0) {
            unsigned prev = __hip_atomic_load(ctr, __ATOMIC_RELAXED, __HIP_MEMORY_SCOPE_AGENT);
            if (lane < 8) lds_store_asm(sync_words + lane, 0u);
            wait_lgkm0();
            __builtin_amdgcn_s_barrier();
            asm volatile("" : "+v"(prev));
            const unsigned call = __builtin_amdgcn_readfirstlane(prev) + 1u;
            const unsigned base = call * 256u;
            __amdgpu_buffer_rsrc_t rd = mkrsrc(p.ws[peer] + (size_t)((call & 1u) * NR + p.rank) * SHARD_BYTES, SHARD_BYTES);
            unsigned* rflag = (unsigned*)(p.ws[peer] + FLAG_OFF) + (p.rank * 16 + idx) * 32;
            const unsigned dofs = (unsigned)((g * 64 + w) * 16) * 1024u + lane * 16u;
            bool bad = false;
            STAMP(1);
#define AG_STAGED(j) do { \
                for (unsigned spin = 1; lds_load(sync_words + (j)) < 4u; ++spin) \
                    if ((spin & 1023u) == 0u && __builtin_amdgcn_s_memrealtime() - t_start > DEADLINE_TICKS) { bad = true; break; } \
                asm volatile("" ::: "memory"); } while (0)
            const int pr = lane & 31, ph = lane >> 5;
            unsigned ua[8];
#pragma unroll
            for (int n = 0; n < 8; ++n) ua[n] = (unsigned)(pr * 512 + ((((n >> 1) * 4 + ph * 2 + (n & 1)) ^ (pr & 15)) << 4));
            u32x4 va[8], vb[8];
            AG_STAGED(0);
#pragma unroll
            for (int n = 0; n < 8; ++n) va[n] = *(const u32x4*)(lds + ua[n]);
#pragma unroll 1
            for (int j = 0; j < 8; ++j) {
                STAMP(3 + j);
#pragma unroll
                for (int n = 0; n < 8; ++n) vb[n] = *(const u32x4*)(lds + j * 16384 + 256 + ua[n]);
#pragma unroll
                for (int n = 0; n < 8; ++n) {
                    wait_vm<WIN - 1>();
                    if (n == WIN - 1 && flag_point(j * 16) && !bad) {
                        if (lane == 0) __hip_atomic_store(rflag, base + (unsigned)(j * 16), __ATOMIC_RELAXED, __HIP_MEMORY_SCOPE_SYSTEM);
                    }
                    __builtin_amdgcn_raw_buffer_store_b128(va[n], rd, dofs + j * (8 * 16 * 1024) + n * 1024, 0, 17);
                }
                if (j < 7) {
                    AG_STAGED(j + 1);
#pragma unroll
                    for (int n = 0; n < 8; ++n) va[n] = *(const u32x4*)(lds + (j + 1) * 16384 + ua[n]);
                }
#pragma unroll
                for (int n = 0; n < 8; ++n) {
                    if (j < 7) wait_vm<WIN - 1>(); else wait_vm<AG_TAIL_WIN - 1>();
                    __builtin_amdgcn_raw_buffer_store_b128(vb[n], rd, dofs + j * (8 * 16 * 1024) + (8 + n) * 1024, 0, 17);
                }
            }
#undef AG_STAGED
            wait_vm<0>();
            if (lane == 0) {
                if (!bad) __hip_atomic_store(rflag, base + 128u, __ATOMIC_RELAXED, __HIP_MEMORY_SCOPE_SYSTEM);
                __hip_atomic_store(ctr, call, __ATOMIC_RELAXED, __HIP_MEMORY_SCOPE_AGENT);
            }
            STAMP(2);
        } else if (wave <= 4) {
            const int v4 = wave - 1;
            const uint8_t* gsrc[4];
#pragma unroll
            for (int u = 0; u < 4; ++u) {
                const int r = v4 * 8 + u * 2 + (lane >> 5);
                gsrc[u] = p.x + (size_t)(g * 256 + r) * KD + w * 512 + (((lane & 31) ^ (r & 15)) << 4);
            }
#define AG_STAGE(j) do { _Pragma("unroll") for (int u = 0; u < 4; ++u) \
                    __builtin_amdgcn_global_load_lds((glb_void*)(gsrc[u] + (size_t)(j) * 32 * KD), \
                                                     (lds_void*)(lds + ((j) * 16 + v4 * 4 + u) * 1024), 16, 0, 0); } while (0)
#define AG_LANDED(j, n) do { wait_vm<n>(); if (lane == 0) lds_add1_asm(sync_words + (j)); } while (0)
            AG_STAGE(0);
            __builtin_amdgcn_s_barrier();
            AG_LANDED(0, 0);
            AG_STAGE(1);
            AG_LANDED(1, 0);
            __builtin_amdgcn_s_sleep(64);
            AG_STAGE(2); AG_STAGE(3);
            AG_LANDED(2, 4); AG_LANDED(3, 0);
            __builtin_amdgcn_s_sleep(127);
            __builtin_amdgcn_s_sleep(127);
            AG_STAGE(4); AG_STAGE(5);
            AG_LANDED(4, 4); AG_LANDED(5, 0);
            __builtin_amdgcn_s_sleep(127);
            AG_STAGE(6); AG_STAGE(7);
            AG_LANDED(6, 4); AG_LANDED(7, 0);
#undef AG_STAGE
            STAMP(1);
#undef AG_LANDED
        } else {
            __builtin_amdgcn_s_barrier();
        }
        STAMP_FLUSH();
        return;
    }

    const int w = wave;
    const int cp = idx & 7;
    char* area = lds + w * 16384;
    if (wave == 0) {
        const unsigned prev = __hip_atomic_load(ctr, __ATOMIC_RELAXED, __HIP_MEMORY_SCOPE_AGENT);
        if (lane == 0) {
            __hip_atomic_store(sync_words, prev + 1u, __ATOMIC_RELAXED, __HIP_MEMORY_SCOPE_WORKGROUP);
            __hip_atomic_store(ctr, prev + 1u, __ATOMIC_RELAXED, __HIP_MEMORY_SCOPE_AGENT);
        }
    }
    if (AG_W_DELAY) __builtin_amdgcn_s_sleep(AG_W_DELAY);
    v8i wf[8][2];
    {
        const uint8_t* gsrc = p.w + (size_t)(w * 512 + (lane >> 2)) * NSH + cp * 64 + (lane & 3) * 16;
        const int grp = lane >> 4, i = lane & 15;
        const unsigned a0 = (unsigned)(((grp >> 1) * 32 + (i >> 1)) * 64 + (grp & 1) * 16 + (i & 1) * 8);
#pragma unroll
        for (int hf = 0; hf < 2; ++hf) {
#pragma unroll
            for (int t = 0; t < 16; ++t)
                __builtin_amdgcn_global_load_lds((glb_void*)(gsrc + (size_t)(hf * 256 + t * 16) * NSH), (lds_void*)(area + t * 1024), 16, 0, 0);
            wait_vm<0>();
#pragma unroll
            for (int s = 0; s < 4; ++s)
#pragma unroll
                for (int t = 0; t < 2; ++t)
#pragma unroll
                    for (int j = 0; j < 4; ++j) {
                        v2i r = __builtin_amdgcn_ds_read_tr8_b64_v2i32(
                            (__attribute__((address_space(3))) v2i*)(area + a0 + (s * 64 + j * 8) * 64 + t * 32));
                        wf[hf * 4 + s][t][2 * j] = r[0];
                        wf[hf * 4 + s][t][2 * j + 1] = r[1];
                    }
            wait_lgkm0();
            __builtin_amdgcn_sched_barrier(0);
        }
    }
    const float scale = p.sx[0] * p.sw[0];
    __builtin_amdgcn_s_barrier();
    asm volatile("" ::: "memory");
    const unsigned call = lds_load(sync_words);
    const unsigned base = call * 256u;
    STAMP(1);
    __amdgpu_buffer_rsrc_t rx = mkrsrc(p.ws[p.rank] + (size_t)((call & 1u) * NR + peer) * SHARD_BYTES, SHARD_BYTES);
    const unsigned* flag = (const unsigned*)(p.ws[p.rank] + FLAG_OFF) + (peer * 16 + g * 8 + w) * 32;
    const unsigned xo = (unsigned)((g * 64 + w) * 16) * 1024u + lane * 16u;
    const int m_w = lane & 31, h_w = lane >> 5;
    const int m_r = w * 4 + (lane >> 4), c_r = lane & 15;
    const unsigned rd_off = (unsigned)(m_r * 256 + ((c_r ^ (m_r & 7)) << 4));
    __amdgpu_buffer_rsrc_t ro = mkrsrc(p.out, (unsigned)(NR * MSH * NSH * 4));
    const unsigned oo = (unsigned)(((peer * MSH + g * 256 + m_r) * NSH + cp * 64 + c_r * 4) * 4);
    bool bad = false, pref = false;
    u32x4 xf[16];

#pragma unroll 1
    for (int j = 0; j < 8; ++j) {
        if (!pref) {
            const unsigned want = base + flag_need(j);
            for (unsigned spin = 1; !bad; ++spin) {
                const unsigned f = __builtin_amdgcn_readfirstlane(__hip_atomic_load(flag, __ATOMIC_RELAXED, AG_POLL_SCOPE));
                if ((int)(f - want) >= 0) break;
                if ((spin & 63u) == 0u && __builtin_amdgcn_s_memrealtime() - t_start > DEADLINE_TICKS) bad = true;
                __builtin_amdgcn_s_sleep(2);
            }
            asm volatile("" ::: "memory");
#pragma unroll
            for (int n = 0; n < 16; ++n)
                xf[n] = __builtin_amdgcn_raw_buffer_load_b128(rx, xo + j * (8 * 16 * 1024) + n * 1024, 0, 16);
        }
        STAMP(2 + 3 * j);
        const unsigned fnext = __hip_atomic_load(flag, __ATOMIC_RELAXED, AG_POLL_SCOPE);
        __builtin_amdgcn_sched_barrier(0);
        f32x16 acc0 = {}, acc1 = {};
#pragma unroll
        for (int s = 0; s < 8; ++s) {
            v8i xa;
            xa[0] = xf[2 * s][0]; xa[1] = xf[2 * s][1]; xa[2] = xf[2 * s][2]; xa[3] = xf[2 * s][3];
            xa[4] = xf[2 * s + 1][0]; xa[5] = xf[2 * s + 1][1]; xa[6] = xf[2 * s + 1][2]; xa[7] = xf[2 * s + 1][3];
            acc0 = __builtin_amdgcn_mfma_scale_f32_32x32x64_f8f6f4(wf[s][0], xa, acc0, 1, 1, 0, 0, 0, 0);
            acc1 = __builtin_amdgcn_mfma_scale_f32_32x32x64_f8f6f4(wf[s][1], xa, acc1, 1, 1, 0, 0, 0, 0);
        }
        __builtin_amdgcn_sched_barrier(0);
        const unsigned fv = __builtin_amdgcn_readfirstlane(fnext);
        asm volatile("" :: "s"(fv));
        pref = j < 7 && (int)(fv - (base + flag_need(j + 1))) >= 0;
        if (pref) {
            asm volatile("" ::: "memory");
#pragma unroll
            for (int n = 0; n < 16; ++n)
                xf[n] = __builtin_amdgcn_raw_buffer_load_b128(rx, xo + (j + 1) * (8 * 16 * 1024) + n * 1024, 0, 16);
        }
        STAMP(3 + 3 * j);
        if (__builtin_expect(bad, 0)) {
            asm volatile("" ::: "memory");
#pragma unroll
            for (int i = 0; i < 16; ++i) acc0[i] = acc1[i] = __builtin_nanf("");
        }
        char* slot = area + (j & 1) * 8192 + m_w * 256;
#pragma unroll
        for (int t = 0; t < 4; ++t) {
            f32x4 v0 = {acc0[4 * t], acc0[4 * t + 1], acc0[4 * t + 2], acc0[4 * t + 3]};
            f32x4 v1 = {acc1[4 * t], acc1[4 * t + 1], acc1[4 * t + 2], acc1[4 * t + 3]};
            *(f32x4*)(slot + (((2 * t + h_w) ^ (m_w & 7)) << 4)) = v0;
            *(f32x4*)(slot + 128 + (((2 * t + h_w) ^ (m_w & 7)) << 4)) = v1;
        }
        wait_lgkm0();
        __builtin_amdgcn_s_barrier();
        asm volatile("" ::: "memory");
        f32x4 sum = *(const f32x4*)(lds + (j & 1) * 8192 + rd_off);
#pragma unroll
        for (int v = 1; v < 8; ++v) sum += *(const f32x4*)(lds + v * 16384 + (j & 1) * 8192 + rd_off);
        f32x4 o;
#pragma unroll
        for (int i = 0; i < 4; ++i) {
            const float y = sum[i] * scale;
            const float yc = fminf(fmaxf(y, -60.0f), 60.0f);
            o[i] = y * __builtin_amdgcn_rcpf(1.0f + __expf(-yc));
        }
        __builtin_amdgcn_raw_buffer_store_b128(__builtin_bit_cast(u32x4, o), ro, oo + j * (32 * NSH * 4), 0, AG_OUT_AUX);
        STAMP(4 + 3 * j);
    }
    STAMP_FLUSH();
}

}

extern "C" void kernel_launch(int rank, int nranks, ncclComm_t comm,
                              hipStream_t stream,
                              void* const* d_in, const int* in_sizes, int n_in,
                              void* d_out, int out_size,
                              const int* in_dtypes, int out_dtype,
                              void* const* d_out_all,
                              void* const* d_ws_all, size_t ws_bytes) {
    (void)comm; (void)in_dtypes; (void)out_dtype; (void)d_out_all;
    if (nranks != NR || (unsigned)rank >= (unsigned)NR || n_in < 4 || ws_bytes < WS_NEED ||
        in_sizes[0] != MSH * KD || in_sizes[1] != KD * NSH || out_size != NR * MSH * NSH) {
        fprintf(stderr, "kernel_launch: unexpected problem shape\n");
        return;
    }
    Params p;
    p.x = (const uint8_t*)d_in[0];
    p.w = (const uint8_t*)d_in[1];
    p.sx = (const float*)d_in[2];
    p.sw = (const float*)d_in[3];
    p.out = (float*)d_out;
    for (int r = 0; r < NR; ++r) p.ws[r] = (char*)d_ws_all[r];
    p.rank = rank;
    hipLaunchKernelGGL(ag_gemm_kernel, dim3(NWG), dim3(NTHREADS), 0, stream, p);
    HIP_CHECK(hipGetLastError());
}
```

```cpp
#include <hip/hip_runtime.h>
#include <rccl/rccl.h>
#include <cstdio>
#include <cstdint>

#define HIP_CHECK(expr) do { hipError_t _e = (expr); if (_e != hipSuccess) { \
    fprintf(stderr, "HIP error %s at %s:%d\n", hipGetErrorString(_e), __FILE__, __LINE__); \
    return; } } while (0)

namespace {

typedef unsigned int u32x4 __attribute__((ext_vector_type(4)));
typedef unsigned int u32x2 __attribute__((ext_vector_type(2)));
typedef int v2i __attribute__((ext_vector_type(2)));
typedef int v8i __attribute__((ext_vector_type(8)));
typedef float f32x4 __attribute__((ext_vector_type(4)));
typedef float f32x16 __attribute__((ext_vector_type(16)));
typedef __attribute__((address_space(3))) unsigned lds_u32;
typedef __attribute__((address_space(3))) void lds_void;
typedef const __attribute__((address_space(1))) void glb_void;

constexpr int NR = 8;
constexpr int KD = 4096;
constexpr int MSH = 512;
constexpr int NSH = 512;
constexpr unsigned SHARD_BYTES = MSH * KD;
constexpr size_t XBUF_BYTES = 2ull * NR * SHARD_BYTES;
constexpr unsigned LL_BYTES = 16 * 8192;
constexpr size_t LL_OFF = XBUF_BYTES;
constexpr size_t FLAG_OFF = LL_OFF + 2ull * NR * LL_BYTES;
constexpr size_t CTR_OFF = FLAG_OFF + NR * 16 * 128;
constexpr size_t WS_NEED = CTR_OFF + 256 * 128;
constexpr unsigned long long DEADLINE_TICKS = 50000000ull;

constexpr int NWG = 256;
constexpr int NTHREADS = 512;
#define AG_WIN 4
#define AG_OUT_AUX 16
#define AG_POLL_SCOPE __HIP_MEMORY_SCOPE_SYSTEM
#define AG_W_DELAY 32
#define AG_LAST_AUX 16
constexpr int WIN = AG_WIN;
constexpr int LDS_SYNC_OFF = 8 * 16384;
constexpr int LDS_BYTES = LDS_SYNC_OFF + 128;

template <int V> struct IntC { static constexpr int value = V; };

struct Params {
    const uint8_t* x;
    const uint8_t* w;
    const float* sx;
    const float* sw;
    float* out;
    char* self;
    char* rot[NR];
    int rank;
};

__device__ __forceinline__ __amdgpu_buffer_rsrc_t mkrsrc(const void* p, unsigned bytes) {
    unsigned long long a = (unsigned long long)p;
    unsigned lo = __builtin_amdgcn_readfirstlane((unsigned)a);
    unsigned hi = __builtin_amdgcn_readfirstlane((unsigned)(a >> 32));
    return __builtin_amdgcn_make_buffer_rsrc((void*)(((unsigned long long)hi << 32) | lo), 0, bytes, 0x00020000);
}
template <int N> __device__ __forceinline__ void wait_vm() { asm volatile("s_waitcnt vmcnt(%0)" :: "i"(N) : "memory"); }
__device__ __forceinline__ void wait_lgkm0() { asm volatile("s_waitcnt lgkmcnt(0)" ::: "memory"); }
__device__ __forceinline__ unsigned lds_load(lds_u32* p) {
    return __builtin_amdgcn_readfirstlane(__hip_atomic_load(p, __ATOMIC_RELAXED, __HIP_MEMORY_SCOPE_WORKGROUP));
}

__device__ __forceinline__ void lds_add1_asm(lds_u32* p) {
    asm volatile("ds_add_u32 %0, %1" :: "v"((unsigned)(uintptr_t)p), "v"(1u) : "memory");
}
__device__ __forceinline__ void lds_store_asm(lds_u32* p, unsigned v) {
    asm volatile("ds_write_b32 %0, %1" :: "v"((unsigned)(uintptr_t)p), "v"(v) : "memory");
}

__device__ __forceinline__ unsigned flag_need(int j) { return j < 2 ? 32u : j < 4 ? 64u : j < 6 ? 96u : 112u; }
__device__ __forceinline__ bool flag_point(int n) { return n == 32 || n == 64 || n == 96 || n == 112; }

#define STAMP(i) do { } while (0)
#define STAMP_FLUSH() do { } while (0)

__global__ void __launch_bounds__(NTHREADS, 2) ag_gemm_kernel(Params p) {
    __shared__ __attribute__((aligned(16))) char lds[LDS_BYTES];
    const int b = blockIdx.x;
    const int lane = threadIdx.x & 63;
    const int wave = __builtin_amdgcn_readfirstlane(threadIdx.x >> 6);
    const int q = b & 7;
    const int idx = b >> 3;
    const int peer = (p.rank + q) & 7;
    const int g = (idx >> 3) & 1;
    const uint8_t* const px = p.x;
    char* const wsp = p.rot[q];
    asm volatile("" :: "s"(px), "s"(wsp), "s"(p.self), "s"(p.rank));
    const unsigned long long t_start = __builtin_amdgcn_s_memrealtime();
    lds_u32* sync_words = (lds_u32*)(lds + LDS_SYNC_OFF);
    unsigned* ctr = (unsigned*)(p.self + CTR_OFF) + b * 32;
    STAMP(0);

    if (__builtin_expect(idx < 16, 1)) {
        const int w = idx & 7;
        if (__builtin_expect(wave == 0, 1)) {
            const int pr = lane & 31, ph = lane >> 5;
            const unsigned prev = __hip_atomic_load(ctr, __ATOMIC_RELAXED, __HIP_MEMORY_SCOPE_AGENT);
            u32x4 va[8], vb[8];
            {
                __amdgpu_buffer_rsrc_t rs = mkrsrc(px, SHARD_BYTES);
                const unsigned so = (unsigned)(g * 256 + pr) * KD + w * 512 + ph * 32;
#pragma unroll
                for (int n = 0; n < 8; ++n) va[n] = __builtin_amdgcn_raw_buffer_load_b128(rs, so + (n >> 1) * 64 + (n & 1) * 16, 0, 0);
            }
            if (lane < 8) lds_store_asm(sync_words + lane, 0u);
            const unsigned call = __builtin_amdgcn_readfirstlane(prev) + 1u;
            const unsigned base = call * 256u;
            __amdgpu_buffer_rsrc_t rd = mkrsrc(wsp + (size_t)((call & 1u) * NR + p.rank) * SHARD_BYTES, SHARD_BYTES);
            unsigned* rflag = (unsigned*)(wsp + FLAG_OFF) + (p.rank * 16 + idx) * 32;
            const unsigned dofs = (unsigned)((g * 64 + w) * 16) * 1024u + lane * 16u;
            bool bad = false;
            STAMP(1);
#pragma unroll
            for (int n = 0; n < 8; ++n) {
                if (n >= WIN) wait_vm<WIN - 1>();
                __builtin_amdgcn_raw_buffer_store_b128(va[n], rd, dofs + n * 1024, 0, 17);
                if (n == 0) STAMP(3);
                if (n == WIN - 1) {
                    wait_lgkm0();
                    __builtin_amdgcn_s_barrier();
                }
            }
#define AG_STAGED(j) do { \
                for (unsigned spin = 1; lds_load(sync_words + (j)) < 4u; ++spin) \
                    if ((spin & 1023u) == 0u && __builtin_amdgcn_s_memrealtime() - t_start > DEADLINE_TICKS) { bad = true; break; } \
                asm volatile("" ::: "memory"); } while (0)
            unsigned ua[8];
#pragma unroll
            for (int n = 0; n < 8; ++n) ua[n] = (unsigned)(pr * 512 + ((((n >> 1) * 4 + ph * 2 + (n & 1)) ^ (pr & 15)) << 4));
            AG_STAGED(0);
#pragma unroll
            for (int n = 0; n < 8; ++n) vb[n] = *(const u32x4*)(lds + 256 + ua[n]);
#pragma unroll 1
            for (int j = 1; j < 8; ++j) {
                AG_STAGED(j);
#pragma unroll
                for (int n = 0; n < 8; ++n) va[n] = *(const u32x4*)(lds + j * 16384 + ua[n]);
#pragma unroll
                for (int n = 0; n < 8; ++n) {
                    wait_vm<WIN - 1>();
                    __builtin_amdgcn_raw_buffer_store_b128(vb[n], rd, dofs + (j - 1) * (8 * 16 * 1024) + (8 + n) * 1024, 0, 17);
                }
                STAMP(3 + j);
#pragma unroll
                for (int n = 0; n < 8; ++n) vb[n] = *(const u32x4*)(lds + j * 16384 + 256 + ua[n]);
#pragma unroll
                for (int n = 0; n < 8; ++n) {
                    wait_vm<WIN - 1>();
                    if (n == WIN - 1 && flag_point(j * 16) && !bad) {
                        if (lane == 0) __hip_atomic_store(rflag, base + (unsigned)(j * 16), __ATOMIC_RELAXED, __HIP_MEMORY_SCOPE_SYSTEM);
                    }
                    __builtin_amdgcn_raw_buffer_store_b128(va[n], rd, dofs + j * (8 * 16 * 1024) + n * 1024, 0, 17);
                }
            }
            {
                __amdgpu_buffer_rsrc_t rl = mkrsrc(wsp + LL_OFF + (size_t)((call & 1u) * NR + p.rank) * LL_BYTES, LL_BYTES);
                const unsigned tagw = (bad ? call + 1u : call) << 24;
#pragma unroll
                for (int n = 0; n < 2; ++n) {
                    wait_vm<WIN - 1>();
                    __builtin_amdgcn_raw_buffer_store_b128(vb[n], rd, dofs + 7 * (8 * 16 * 1024) + (8 + n) * 1024, 0, 17);
                }
                u32x2 e;
                e[0] = (vb[2][3] >> 24) | ((vb[3][3] >> 24) << 8) | ((vb[4][3] >> 24) << 16) | (vb[5][3] & 0xff000000u);
                e[1] = (vb[6][3] >> 24) | ((vb[7][3] >> 24) << 8) | tagw;
                wait_vm<WIN - 1>();
                __builtin_amdgcn_raw_buffer_store_b64(e, rl, (unsigned)(idx * 8 + 6) * 1024u + lane * 8u, 0, 17);
#pragma unroll
                for (int t = 0; t < 6; ++t) {
                    wait_vm<WIN - 1>();
                    if (t == WIN - 2 && !bad) {
                        if (lane == 0) __hip_atomic_store(rflag, base + 122u, __ATOMIC_RELAXED, __HIP_MEMORY_SCOPE_SYSTEM);
                    }
                    u32x4 u = vb[2 + t];
                    u[3] = (u[3] & 0x00ffffffu) | tagw;
                    __builtin_amdgcn_raw_buffer_store_b128(u, rl, (unsigned)(idx * 8 + t) * 1024u + lane * 16u, 0, 17);
                }
            }
#undef AG_STAGED
            if (lane == 0) __hip_atomic_store(ctr, call, __ATOMIC_RELAXED, __HIP_MEMORY_SCOPE_AGENT);
            STAMP(2);
        } else if (wave <= 4) {
            const int v4 = wave - 1;
            const uint8_t* gsrc[4];
#pragma unroll
            for (int u = 0; u < 4; ++u) {
                const int r = v4 * 8 + u * 2 + (lane >> 5);
                gsrc[u] = px + (size_t)(g * 256 + r) * KD + w * 512 + (((lane & 31) ^ (r & 15)) << 4);
            }
#define AG_STAGE(j) do { _Pragma("unroll") for (int u = 0; u < 4; ++u) \
                    __builtin_amdgcn_global_load_lds((glb_void*)(gsrc[u] + (size_t)(j) * 32 * KD), \
                                                     (lds_void*)(lds + ((j) * 16 + v4 * 4 + u) * 1024), 16, 0, 0); } while (0)
#define AG_LANDED(j, n) do { wait_vm<n>(); if (lane == 0) lds_add1_asm(sync_words + (j)); } while (0)
            AG_STAGE(0);
            __builtin_amdgcn_s_barrier();
            AG_LANDED(0, 0);
            AG_STAGE(1);
            AG_LANDED(1, 0);
            __builtin_amdgcn_s_sleep(64);
            AG_STAGE(2); AG_STAGE(3);
            AG_LANDED(2, 4); AG_LANDED(3, 0);
            __builtin_amdgcn_s_sleep(127);
            __builtin_amdgcn_s_sleep(127);
            AG_STAGE(4); AG_STAGE(5);
            AG_LANDED(4, 4); AG_LANDED(5, 0);
            __builtin_amdgcn_s_sleep(127);
            AG_STAGE(6); AG_STAGE(7);
            AG_LANDED(6, 4); AG_LANDED(7, 0);
#undef AG_STAGE
            STAMP(1);
#undef AG_LANDED
        } else {
            __builtin_amdgcn_s_barrier();
        }
        STAMP_FLUSH();
        return;
    }

    const int w = wave;
    const int cp = idx & 7;
    char* area = lds + w * 16384;
    if (wave == 0) {
        const unsigned prev = __hip_atomic_load(ctr, __ATOMIC_RELAXED, __HIP_MEMORY_SCOPE_AGENT);
        if (lane == 0) {
            __hip_atomic_store(sync_words, prev + 1u, __ATOMIC_RELAXED, __HIP_MEMORY_SCOPE_WORKGROUP);
            __hip_atomic_store(ctr, prev + 1u, __ATOMIC_RELAXED, __HIP_MEMORY_SCOPE_AGENT);
        }
    }
    if (AG_W_DELAY) __builtin_amdgcn_s_sleep(AG_W_DELAY);
    v8i wf[8][2];
    {
        const uint8_t* gsrc = p.w + (size_t)(w * 512 + (lane >> 2)) * NSH + cp * 64 + (lane & 3) * 16;
        const int grp = lane >> 4, i = lane & 15;
        const unsigned a0 = (unsigned)(((grp >> 1) * 32 + (i >> 1)) * 64 + (grp & 1) * 16 + (i & 1) * 8);
#pragma unroll
        for (int hf = 0; hf < 2; ++hf) {
#pragma unroll
            for (int t = 0; t < 16; ++t)
                __builtin_amdgcn_global_load_lds((glb_void*)(gsrc + (size_t)(hf * 256 + t * 16) * NSH), (lds_void*)(area + t * 1024), 16, 0, 0);
            wait_vm<0>();
#pragma unroll
            for (int s = 0; s < 4; ++s)
#pragma unroll
                for (int t = 0; t < 2; ++t)
#pragma unroll
                    for (int j = 0; j < 4; ++j) {
                        v2i r = __builtin_amdgcn_ds_read_tr8_b64_v2i32(
                            (__attribute__((address_space(3))) v2i*)(area + a0 + (s * 64 + j * 8) * 64 + t * 32));
                        wf[hf * 4 + s][t][2 * j] = r[0];
                        wf[hf * 4 + s][t][2 * j + 1] = r[1];
                    }
            wait_lgkm0();
            __builtin_amdgcn_sched_barrier(0);
        }
    }
    const float scale = p.sx[0] * p.sw[0];
    __builtin_amdgcn_s_barrier();
    asm volatile("" ::: "memory");
    const unsigned call = lds_load(sync_words);
    const unsigned base = call * 256u;
    STAMP(1);
    __amdgpu_buffer_rsrc_t rx = mkrsrc(p.self + (size_t)((call & 1u) * NR + peer) * SHARD_BYTES, SHARD_BYTES);
    const unsigned* flag = (const unsigned*)(p.self + FLAG_OFF) + (peer * 16 + g * 8 + w) * 32;
    const unsigned xo = (unsigned)((g * 64 + w) * 16) * 1024u + lane * 16u;
    const int m_w = lane & 31, h_w = lane >> 5;
    const int m_r = w * 4 + (lane >> 4), c_r = lane & 15;
    const unsigned rd_off = (unsigned)(m_r * 256 + ((c_r ^ (m_r & 7)) << 4));
    __amdgpu_buffer_rsrc_t ro = mkrsrc(p.out, (unsigned)(NR * MSH * NSH * 4));
    const unsigned oo = (unsigned)(((peer * MSH + g * 256 + m_r) * NSH + cp * 64 + c_r * 4) * 4);
    bool bad = false, pref = false;
    u32x4 xf[16];

#define AG_POLL(want) do { \
        for (unsigned spin = 1; !bad; ++spin) { \
            const unsigned f = __builtin_amdgcn_readfirstlane(__hip_atomic_load(flag, __ATOMIC_RELAXED, AG_POLL_SCOPE)); \
            if ((int)(f - (want)) >= 0) break; \
            if ((spin & 63u) == 0u && __builtin_amdgcn_s_memrealtime() - t_start > DEADLINE_TICKS) bad = true; \
            __builtin_amdgcn_s_sleep(2); \
        } \
        asm volatile("" ::: "memory"); } while (0)
#define AG_MFMA(s, lo, hi) do { \
        v8i xa; \
        xa[0] = (lo)[0]; xa[1] = (lo)[1]; xa[2] = (lo)[2]; xa[3] = (lo)[3]; \
        xa[4] = (hi)[0]; xa[5] = (hi)[1]; xa[6] = (hi)[2]; xa[7] = (hi)[3]; \
        acc0 = __builtin_amdgcn_mfma_scale_f32_32x32x64_f8f6f4(wf[s][0], xa, acc0, 1, 1, 0, 0, 0, 0); \
        acc1 = __builtin_amdgcn_mfma_scale_f32_32x32x64_f8f6f4(wf[s][1], xa, acc1, 1, 1, 0, 0, 0, 0); } while (0)
    auto finish = [&](int j, f32x16& acc0, f32x16& acc1, auto aux) {
        if (__builtin_expect(bad, 0)) {
            asm volatile("" ::: "memory");
#pragma unroll
            for (int i = 0; i < 16; ++i) acc0[i] = acc1[i] = __builtin_nanf("");
        }
        char* slot = area + (j & 1) * 8192 + m_w * 256;
#pragma unroll
        for (int t = 0; t < 4; ++t) {
            f32x4 v0 = {acc0[4 * t], acc0[4 * t + 1], acc0[4 * t + 2], acc0[4 * t + 3]};
            f32x4 v1 = {acc1[4 * t], acc1[4 * t + 1], acc1[4 * t + 2], acc1[4 * t + 3]};
            *(f32x4*)(slot + (((2 * t + h_w) ^ (m_w & 7)) << 4)) = v0;
            *(f32x4*)(slot + 128 + (((2 * t + h_w) ^ (m_w & 7)) << 4)) = v1;
        }
        wait_lgkm0();
        __builtin_amdgcn_s_barrier();
        asm volatile("" ::: "memory");
        f32x4 part[8];
#pragma unroll
        for (int v = 0; v < 8; ++v) part[v] = *(const f32x4*)(lds + v * 16384 + (j & 1) * 8192 + rd_off);
        const f32x4 sum = ((part[0] + part[1]) + (part[2] + part[3])) + ((part[4] + part[5]) + (part[6] + part[7]));
        f32x4 o;
#pragma unroll
        for (int i = 0; i < 4; ++i) {
            const float y = sum[i] * scale;
            const float yc = fminf(fmaxf(y, -60.0f), 60.0f);
            o[i] = y * __builtin_amdgcn_rcpf(1.0f + __expf(-yc));
        }
        __builtin_amdgcn_raw_buffer_store_b128(__builtin_bit_cast(u32x4, o), ro, oo + j * (32 * NSH * 4), 0, decltype(aux)::value);
    };

#pragma unroll 1
    for (int j = 0; j < 7; ++j) {
        if (!pref) {
            AG_POLL(base + flag_need(j));
#pragma unroll
            for (int n = 0; n < 16; ++n)
                xf[n] = __builtin_amdgcn_raw_buffer_load_b128(rx, xo + j * (8 * 16 * 1024) + n * 1024, 0, 16);
        }
        STAMP(2 + 3 * j);
        const unsigned fnext = __hip_atomic_load(flag, __ATOMIC_RELAXED, AG_POLL_SCOPE);
        __builtin_amdgcn_sched_barrier(0);
        f32x16 acc0 = {}, acc1 = {};
#pragma unroll
        for (int s = 0; s < 8; ++s) AG_MFMA(s, xf[2 * s], xf[2 * s + 1]);
        __builtin_amdgcn_sched_barrier(0);
        const unsigned fv = __builtin_amdgcn_readfirstlane(fnext);
        asm volatile("" :: "s"(fv));
        pref = j < 6 && (int)(fv - (base + flag_need(j + 1))) >= 0;
        if (pref) {
            asm volatile("" ::: "memory");
#pragma unroll
            for (int n = 0; n < 16; ++n)
                xf[n] = __builtin_amdgcn_raw_buffer_load_b128(rx, xo + (j + 1) * (8 * 16 * 1024) + n * 1024, 0, 16);
        }
        STAMP(3 + 3 * j);
        finish(j, acc0, acc1, IntC<AG_OUT_AUX>{});
        STAMP(4 + 3 * j);
    }
    {
        __amdgpu_buffer_rsrc_t rl = mkrsrc(p.self + LL_OFF + (size_t)((call & 1u) * NR + peer) * LL_BYTES, LL_BYTES);
        const unsigned lo = (unsigned)((g * 8 + w) * 8) * 1024u + lane * 16u;
        const unsigned eo = (unsigned)((g * 8 + w) * 8 + 6) * 1024u + lane * 8u;
        const unsigned tagw = call << 24;
        u32x4 ll[6];
        u32x2 le;
        AG_POLL(base + 122u);
        STAMP(23);
#pragma unroll
        for (int n = 0; n < 10; ++n)
            xf[n] = __builtin_amdgcn_raw_buffer_load_b128(rx, xo + 7 * (8 * 16 * 1024) + n * 1024, 0, 16);
        le = __builtin_amdgcn_raw_buffer_load_b64(rl, eo, 0, 17);
#pragma unroll
        for (int t = 0; t < 6; ++t) ll[t] = __builtin_amdgcn_raw_buffer_load_b128(rl, lo + t * 1024, 0, 17);
        __builtin_amdgcn_sched_barrier(0);
        f32x16 acc0 = {}, acc1 = {};
#pragma unroll
        for (int s = 0; s < 5; ++s) AG_MFMA(s, xf[2 * s], xf[2 * s + 1]);
        __builtin_amdgcn_sched_barrier(0);
        asm volatile("" : "+v"(acc0), "+v"(acc1));
#define AG_LL_STEP(s) do { \
            for (unsigned spin = 1;; ++spin) { \
                unsigned df = (ll[2 * (s)][3] ^ tagw) | (ll[2 * (s) + 1][3] ^ tagw); \
                if ((s) == 0) df |= le[1] ^ tagw; \
                if (__all((df >> 24) == 0u) || bad) break; \
                if ((spin & 63u) == 0u && __builtin_amdgcn_s_memrealtime() - t_start > DEADLINE_TICKS) bad = true; \
                if ((s) == 0) le = __builtin_amdgcn_raw_buffer_load_b64(rl, eo, 0, 17); \
                _Pragma("unroll") for (int t = 2 * (s); t < 6; ++t) ll[t] = __builtin_amdgcn_raw_buffer_load_b128(rl, lo + t * 1024, 0, 17); \
            } \
            u32x4 a = ll[2 * (s)], c = ll[2 * (s) + 1]; \
            a[3] = (a[3] & 0x00ffffffu) | ((le[(s) >> 1] >> (16 * ((s) & 1))) << 24); \
            c[3] = (c[3] & 0x00ffffffu) | ((le[(s) >> 1] >> (16 * ((s) & 1) + 8)) << 24); \
            if ((s) == 2) STAMP(25); \
            AG_MFMA(5 + (s), a, c); \
            asm volatile("" : "+v"(acc0), "+v"(acc1)); } while (0)
        AG_LL_STEP(0);
        AG_LL_STEP(1);
        AG_LL_STEP(2);
#undef AG_LL_STEP
        STAMP(24);
        finish(7, acc0, acc1, IntC<AG_LAST_AUX>{});
        STAMP(26);
    }
#undef AG_POLL
#undef AG_MFMA
    STAMP_FLUSH();
}

}

extern "C" void kernel_launch(int rank, int nranks, ncclComm_t comm,
                              hipStream_t stream,
                              void* const* d_in, const int* in_sizes, int n_in,
                              void* d_out, int out_size,
                              const int* in_dtypes, int out_dtype,
                              void* const* d_out_all,
                              void* const* d_ws_all, size_t ws_bytes) {
    (void)comm; (void)in_dtypes; (void)out_dtype; (void)d_out_all;
    if (nranks != NR || (unsigned)rank >= (unsigned)NR || n_in < 4 || ws_bytes < WS_NEED ||
        in_sizes[0] != MSH * KD || in_sizes[1] != KD * NSH || out_size != NR * MSH * NSH) {
        fprintf(stderr, "kernel_launch: unexpected problem shape\n");
        return;
    }
    Params p;
    p.x = (const uint8_t*)d_in[0];
    p.w = (const uint8_t*)d_in[1];
    p.sx = (const float*)d_in[2];
    p.sw = (const float*)d_in[3];
    p.out = (float*)d_out;
    p.self = (char*)d_ws_all[rank];
    for (int r = 0; r < NR; ++r) p.rot[r] = (char*)d_ws_all[(rank + r) % NR];
    p.rank = rank;
    hipLaunchKernelGGL(ag_gemm_kernel, dim3(NWG), dim3(NTHREADS), 0, stream, p);
    HIP_CHECK(hipGetLastError());
}
```

```cpp
#include <hip/hip_runtime.h>
#include <rccl/rccl.h>
#include <cstdio>
#include <cstdint>

#define HIP_CHECK(expr) do { hipError_t _e = (expr); if (_e != hipSuccess) { \
    fprintf(stderr, "HIP error %s at %s:%d\n", hipGetErrorString(_e), __FILE__, __LINE__); \
    return; } } while (0)

namespace {

typedef unsigned int u32x4 __attribute__((ext_vector_type(4)));
typedef unsigned int u32x2 __attribute__((ext_vector_type(2)));
typedef int v2i __attribute__((ext_vector_type(2)));
typedef int v8i __attribute__((ext_vector_type(8)));
typedef float f32x4 __attribute__((ext_vector_type(4)));
typedef float f32x16 __attribute__((ext_vector_type(16)));
typedef __attribute__((address_space(3))) unsigned lds_u32;
typedef __attribute__((address_space(3))) void lds_void;
typedef const __attribute__((address_space(1))) void glb_void;

constexpr int NR = 8;
constexpr int KD = 4096;
constexpr int MSH = 512;
constexpr int NSH = 512;
constexpr unsigned SHARD_BYTES = MSH * KD;
constexpr unsigned R_W = SHARD_BYTES;
constexpr unsigned R_LL = R_W + 2 * KD * 64;
constexpr unsigned REGION = 3u << 20;
constexpr unsigned PAR_BYTES = NR * REGION;
constexpr size_t FLAG_OFF = 2ull * PAR_BYTES;
constexpr size_t DONE_OFF = FLAG_OFF + NR * 16 * 128;
constexpr size_t CTR_OFF = DONE_OFF + NR * 128;
constexpr size_t WS_NEED = CTR_OFF + 256 * 128;
static_assert(R_LL + 16 * 8192 <= REGION, "region layout");
constexpr unsigned long long DEADLINE_TICKS = 50000000ull;

constexpr int NWG = 256;
constexpr int NTHREADS = 512;
#define AG_WIN 4
#define AG_W_DELAY 32
constexpr int WIN = AG_WIN;
constexpr int LDS_SYNC_OFF = 8 * 16384;
constexpr int LDS_BYTES = LDS_SYNC_OFF + 128;

struct Params {
    const uint8_t* x;
    const uint8_t* w;
    const float* sx;
    const float* sw;
    char* ws[NR];
    float* out[NR];
    int rank;
};

__device__ __forceinline__ __amdgpu_buffer_rsrc_t mkrsrc(const void* p, unsigned bytes) {
    unsigned long long a = (unsigned long long)p;
    unsigned lo = __builtin_amdgcn_readfirstlane((unsigned)a);
    unsigned hi = __builtin_amdgcn_readfirstlane((unsigned)(a >> 32));
    return __builtin_amdgcn_make_buffer_rsrc((void*)(((unsigned long long)hi << 32) | lo), 0, bytes, 0x00020000);
}
template <int N> __device__ __forceinline__ void wait_vm() { asm volatile("s_waitcnt vmcnt(%0)" :: "i"(N) : "memory"); }
__device__ __forceinline__ void wait_lgkm0() { asm volatile("s_waitcnt lgkmcnt(0)" ::: "memory"); }
__device__ __forceinline__ unsigned lds_load(lds_u32* p) {
    return __builtin_amdgcn_readfirstlane(__hip_atomic_load(p, __ATOMIC_RELAXED, __HIP_MEMORY_SCOPE_WORKGROUP));
}
__device__ __forceinline__ void lds_add1_asm(lds_u32* p) {
    asm volatile("ds_add_u32 %0, %1" :: "v"((unsigned)(uintptr_t)p), "v"(1u) : "memory");
}
__device__ __forceinline__ void lds_store_asm(lds_u32* p, unsigned v) {
    asm volatile("ds_write_b32 %0, %1" :: "v"((unsigned)(uintptr_t)p), "v"(v) : "memory");
}

#define STAMP(i) do { } while (0)
#define STAMP_FLUSH() do { } while (0)

struct Step { unsigned xoff, foff, need, orow; };
__device__ __forceinline__ Step make_step(int src, int g, int j, unsigned need) {
    Step s;
    s.xoff = (unsigned)src * REGION + (unsigned)g * (1u << 20) + (unsigned)j * (1u << 17);
    s.foff = (unsigned)(src * 16 + g * 8) * 128u;
    s.need = need;
    s.orow = (unsigned)(src * MSH + g * 256 + j * 32) * (NSH * 4u);
    return s;
}
__device__ __forceinline__ unsigned pack_step(int rel, int g, int j, int need) { return (unsigned)(rel | g << 3 | j << 4 | (need >> 4) << 7); }
__device__ __forceinline__ unsigned plan_step(int q, int c, int i) {
    if (q < 4) {
        const int g = c >> 3;
        const int i6 = q == 0 ? 8 : 2, i7 = q == 0 ? 9 : 6;
        if (i == i6) return pack_step(4 + q, g, 6, 48);
        if (i == i7) return pack_step(4 + q, g, 7, 64);
        const int j = i - (i > i6) - (i > i7);
        return pack_step(q, g, j, (j < 4 || q == 0) ? 32 * (j / 2 + 1) : 16 * (j + 1));
    }
    const int e = (c & 7) + 8 * i;
    if (e < 12) return pack_step(0, e & 1, e >> 1, 32 * ((e >> 2) + 1));
    if (e < 24) { const int f = e - 12; return pack_step(1 + (f >> 2), f & 1, (f >> 1) & 1, 32); }
    if (e < 30) { const int f = e - 24; return pack_step(4 + ((q + 1 + (f >> 1)) & 3), f & 1, 6, 48); }
    if (e < 42) { const int f = e - 30; return pack_step(1 + (f >> 2), f & 1, 2 + ((f >> 1) & 1), 64); }
    const int f = e < 48 ? e - 42 : e - 48, j = e < 48 ? 4 : 5;
    return pack_step(1 + ((f >> 1) & 3), f & 1, j, 16 * (j + 1));
}

__global__ void __launch_bounds__(NTHREADS, 2) ag_gemm_kernel(Params p) {
    __shared__ __attribute__((aligned(16))) char lds[LDS_BYTES];
    const int b = blockIdx.x;
    const int lane = threadIdx.x & 63;
    const int wave = __builtin_amdgcn_readfirstlane(threadIdx.x >> 6);
    const int q = b & 7;
    const int idx = b >> 3;
    const int g = (idx >> 3) & 1;
    const uint8_t* const px = p.x;
    const uint8_t* const pw = p.w;
    char* const self = p.ws[0];
    char* const wsp = p.ws[q];
    const int rank = p.rank;
    asm volatile("" :: "s"(px), "s"(pw), "s"(wsp), "s"(self), "s"(rank));
    const int peer = rank ^ q;
    const unsigned long long t_start = __builtin_amdgcn_s_memrealtime();
    lds_u32* sync_words = (lds_u32*)(lds + LDS_SYNC_OFF);
    unsigned* ctr = (unsigned*)(self + CTR_OFF) + b * 32;
    STAMP(0);

    if (__builtin_expect(idx < 16, 1)) {
        const int w = idx & 7;
        const bool cross = q >= 4;
        const int wcol = (peer & 3) * 128 + g * 64;
        if (__builtin_expect(wave == 0, 1)) {
            const int pr = lane & 31, ph = lane >> 5;
            const unsigned prev = __hip_atomic_load(ctr, __ATOMIC_RELAXED, __HIP_MEMORY_SCOPE_AGENT);
            u32x4 va[8], vb[8];
            {
                __amdgpu_buffer_rsrc_t rs = mkrsrc(cross ? pw : px, SHARD_BYTES);
                const unsigned so = cross ? (unsigned)(w * 512 + (lane >> 2)) * NSH + wcol + (lane & 3) * 16
                                          : (unsigned)(g * 256 + pr) * KD + w * 512 + ph * 32;
                const unsigned sa = cross ? 16384u : 64u, sb = cross ? 8192u : 16u;
#pragma unroll
                for (int n = 0; n < 8; ++n) va[n] = __builtin_amdgcn_raw_buffer_load_b128(rs, so, (n >> 1) * sa + (n & 1) * sb, 0);
            }
            if (lane < 8) lds_store_asm(sync_words + lane, 0u);
            const unsigned call = __builtin_amdgcn_readfirstlane(prev) + 1u;
            const unsigned base = call * 256u;
            __amdgpu_buffer_rsrc_t rd = mkrsrc(wsp + (size_t)((call & 1u) * NR + rank) * REGION, REGION);
            unsigned* rflag = (unsigned*)(wsp + FLAG_OFF) + (rank * 16 + idx) * 32;
            const unsigned xs = (unsigned)((g * 64 + w) * 16) * 1024u + lane * 16u;
            const unsigned wsl = R_W + (unsigned)((g * 8 + w) * 32) * 1024u + lane * 16u;
            const int nseg = q == 0 ? 8 : cross ? 4 : 7;
            const unsigned fmask = q == 0 ? 0x54u : cross ? 0x0eu : 0x74u;
            bool bad = false;
            unsigned dprev = cross ? wsl : xs;
            STAMP(1);
#pragma unroll
            for (int n = 0; n < 8; ++n) {
                if (n >= WIN) wait_vm<WIN - 1>();
                __builtin_amdgcn_raw_buffer_store_b128(va[n], rd, dprev + n * 1024, 0, 17);
                if (n == 0) STAMP(3);
                if (n == WIN - 1) {
                    if (lane == 0) lds_store_asm(sync_words + 8, call);
                    wait_lgkm0();
                    __builtin_amdgcn_s_barrier();
                }
            }
#define AG_STAGED(j) do { \
                for (unsigned spin = 1; lds_load(sync_words + (j)) < 4u; ++spin) \
                    if ((spin & 1023u) == 0u && __builtin_amdgcn_s_memrealtime() - t_start > DEADLINE_TICKS) { bad = true; break; } \
                asm volatile("" ::: "memory"); } while (0)
            unsigned ua[8];
#pragma unroll
            for (int n = 0; n < 8; ++n) ua[n] = (unsigned)(pr * 512 + ((((n >> 1) * 4 + ph * 2 + (n & 1)) ^ (pr & 15)) << 4));
            AG_STAGED(0);
#pragma unroll
            for (int n = 0; n < 8; ++n) vb[n] = *(const u32x4*)(lds + 256 + ua[n]);
#pragma unroll 1
            for (int s = 1; s < nseg; ++s) {
                const int jt = cross ? s + 4 : (q != 0 && s == 6) ? 7 : s;
                const unsigned dcur = (cross && s == 1) ? wsl + 16384u : xs + (unsigned)jt * (1u << 17);
                AG_STAGED(s);
#pragma unroll
                for (int n = 0; n < 8; ++n) va[n] = *(const u32x4*)(lds + s * 16384 + ua[n]);
#pragma unroll
                for (int n = 0; n < 8; ++n) {
                    wait_vm<WIN - 1>();
                    __builtin_amdgcn_raw_buffer_store_b128(vb[n], rd, dprev + (8 + n) * 1024, 0, 17);
                }
                STAMP(3 + s);
#pragma unroll
                for (int n = 0; n < 8; ++n) vb[n] = *(const u32x4*)(lds + s * 16384 + 256 + ua[n]);
#pragma unroll
                for (int n = 0; n < 8; ++n) {
                    wait_vm<WIN - 1>();
                    if (n == WIN - 1 && ((fmask >> s) & 1u) && !bad) {
                        if (lane == 0) __hip_atomic_store(rflag, base + (unsigned)(s * 16), __ATOMIC_RELAXED, __HIP_MEMORY_SCOPE_SYSTEM);
                    }
                    __builtin_amdgcn_raw_buffer_store_b128(va[n], rd, dcur + n * 1024, 0, 17);
                }
                dprev = dcur;
            }
#pragma unroll
            for (int n = 0; n < 2; ++n) {
                wait_vm<WIN - 1>();
                __builtin_amdgcn_raw_buffer_store_b128(vb[n], rd, dprev + (8 + n) * 1024, 0, 17);
            }
            if (q != 0 && !cross) {
                const unsigned tagw = (bad ? call + 1u : call) << 24;
                u32x2 e;
                e[0] = (vb[2][3] >> 24) | ((vb[3][3] >> 24) << 8) | ((vb[4][3] >> 24) << 16) | (vb[5][3] & 0xff000000u);
                e[1] = (vb[6][3] >> 24) | ((vb[7][3] >> 24) << 8) | tagw;
                wait_vm<WIN - 1>();
                __builtin_amdgcn_raw_buffer_store_b64(e, rd, R_LL + (unsigned)(idx * 8 + 6) * 1024u + lane * 8u, 0, 17);
#pragma unroll
                for (int t = 0; t < 6; ++t) {
                    wait_vm<WIN - 1>();
                    if (t == WIN - 2 && !bad) {
                        if (lane == 0) __hip_atomic_store(rflag, base + 106u, __ATOMIC_RELAXED, __HIP_MEMORY_SCOPE_SYSTEM);
                    }
                    u32x4 u = vb[2 + t];
                    u[3] = (u[3] & 0x00ffffffu) | tagw;
                    __builtin_amdgcn_raw_buffer_store_b128(u, rd, R_LL + (unsigned)(idx * 8 + t) * 1024u + lane * 16u, 0, 17);
                }
            } else {
#pragma unroll
                for (int n = 2; n < 8; ++n) {
                    wait_vm<WIN - 1>();
                    __builtin_amdgcn_raw_buffer_store_b128(vb[n], rd, dprev + (8 + n) * 1024, 0, 17);
                }
                wait_vm<0>();
                if (lane == 0 && !bad) __hip_atomic_store(rflag, base + (unsigned)(nseg * 16), __ATOMIC_RELAXED, __HIP_MEMORY_SCOPE_SYSTEM);
            }
#undef AG_STAGED
            if (lane == 0) __hip_atomic_store(ctr, call, __ATOMIC_RELAXED, __HIP_MEMORY_SCOPE_AGENT);
            STAMP(2);
        } else if (wave <= 4) {
            const int v4 = wave - 1;
            const uint8_t* gsrc[4];
            const uint8_t* wsrc[4];
#pragma unroll
            for (int u = 0; u < 4; ++u) {
                const int r = v4 * 8 + u * 2 + (lane >> 5);
                const int ch = (lane & 31) ^ (r & 15);
                gsrc[u] = px + (size_t)(g * 256 + r) * KD + w * 512 + (ch << 4);
                const int un = (ch >> 4) * 8 + ((ch >> 2) & 3) * 2 + (ch & 1), l = r + 32 * ((ch >> 1) & 1);
                wsrc[u] = pw + (size_t)(w * 512 + un * 16 + (l >> 2)) * NSH + wcol + (l & 3) * 16;
            }
#define AG_STAGE(src, off, slot) do { _Pragma("unroll") for (int u = 0; u < 4; ++u) \
                    __builtin_amdgcn_global_load_lds((glb_void*)(src[u] + (size_t)(off)), \
                                                     (lds_void*)(lds + ((slot) * 16 + v4 * 4 + u) * 1024), 16, 0, 0); } while (0)
#define AG_STAGE_X(j, slot) AG_STAGE(gsrc, (j) * 32 * KD, slot)
#define AG_LANDED(slot, n) do { wait_vm<n>(); if (lane == 0) lds_add1_asm(sync_words + (slot)); } while (0)
            if (cross) {
                AG_STAGE(wsrc, 0, 0);
                __builtin_amdgcn_s_barrier();
                AG_LANDED(0, 0);
                AG_STAGE(wsrc, 256 * NSH, 1);
                AG_LANDED(1, 0);
                __builtin_amdgcn_s_sleep(64);
                AG_STAGE_X(6, 2); AG_STAGE_X(7, 3);
                AG_LANDED(2, 4); AG_LANDED(3, 0);
            } else {
                AG_STAGE_X(0, 0);
                __builtin_amdgcn_s_barrier();
                AG_LANDED(0, 0);
                AG_STAGE_X(1, 1);
                AG_LANDED(1, 0);
                __builtin_amdgcn_s_sleep(64);
                AG_STAGE_X(2, 2); AG_STAGE_X(3, 3);
                AG_LANDED(2, 4); AG_LANDED(3, 0);
                if (q != 0) {
                    __builtin_amdgcn_s_sleep(127);
                    __builtin_amdgcn_s_sleep(127);
                }
                AG_STAGE_X(4, 4); AG_STAGE_X(5, 5);
                AG_LANDED(4, 4); AG_LANDED(5, 0);
                if (q != 0) {
                    __builtin_amdgcn_s_sleep(127);
                    AG_STAGE_X(7, 6);
                    AG_LANDED(6, 0);
                } else {
                    AG_STAGE_X(6, 6); AG_STAGE_X(7, 7);
                    AG_LANDED(6, 4); AG_LANDED(7, 0);
                }
            }
#undef AG_STAGE
#undef AG_STAGE_X
#undef AG_LANDED
            STAMP(1);
        } else {
            __builtin_amdgcn_s_barrier();
            if (b == 0 && wave == 7) {
                asm volatile("" ::: "memory");
                const unsigned call = lds_load(sync_words + 8);
                const unsigned* done = (const unsigned*)(self + DONE_OFF) + (rank ^ (4 + (lane >> 4))) * 32 + (lane & 15);
                for (unsigned spin = 1;; ++spin) {
                    const unsigned f = __hip_atomic_load(done, __ATOMIC_RELAXED, __HIP_MEMORY_SCOPE_SYSTEM);
                    if (__all((int)(f - call) >= 0)) break;
                    if ((spin & 63u) == 0u && __builtin_amdgcn_s_memrealtime() - t_start > DEADLINE_TICKS) {
                        p.out[0][(size_t)rank * MSH * NSH + lane] = __builtin_nanf("");
                        break;
                    }
                    __builtin_amdgcn_s_sleep(2);
                }
                STAMP(2);
            }
        }
        STAMP_FLUSH();
        return;
    }

    const int w = wave;
    const int c = idx - 16;
    const bool cross = q >= 4;
    char* area = lds + w * 16384;
    if (wave == 0) {
        const unsigned prev = __hip_atomic_load(ctr, __ATOMIC_RELAXED, __HIP_MEMORY_SCOPE_AGENT);
        if (lane == 0) {
            __hip_atomic_store(sync_words, prev + 1u, __ATOMIC_RELAXED, __HIP_MEMORY_SCOPE_WORKGROUP);
            __hip_atomic_store(ctr, prev + 1u, __ATOMIC_RELAXED, __HIP_MEMORY_SCOPE_AGENT);
        }
        wait_lgkm0();
    }
    const float scale = p.sx[0] * p.sw[0];
    __builtin_amdgcn_s_barrier();
    asm volatile("" ::: "memory");
    const unsigned call = lds_load(sync_words);
    const unsigned base = call * 256u;
    const char* const par = self + (size_t)(call & 1u) * PAR_BYTES;
    const char* const flags = self + FLAG_OFF + w * 128;
    bool bad = false;

#define AG_POLL(fptr, want) do { \
        for (unsigned spin = 1; !bad; ++spin) { \
            const unsigned f = __builtin_amdgcn_readfirstlane(__hip_atomic_load((const unsigned*)(fptr), __ATOMIC_RELAXED, __HIP_MEMORY_SCOPE_SYSTEM)); \
            if ((int)(f - (want)) >= 0) break; \
            if ((spin & 63u) == 0u && __builtin_amdgcn_s_memrealtime() - t_start > DEADLINE_TICKS) bad = true; \
            __builtin_amdgcn_s_sleep(2); \
        } \
        asm volatile("" ::: "memory"); } while (0)

    const int nstep = q == 0 ? 10 : !cross ? 8 : (c & 7) < 6 ? 7 : 6;
    unsigned long long plan_lo = 0, plan_hi = 0;
#pragma unroll 1
    for (int i = 9; i >= 0; --i) {
        plan_hi = (plan_hi << 12) | (plan_lo >> 48);
        plan_lo = ((plan_lo << 12) | plan_step(q, c, i < nstep ? i : 0)) & 0xfffffffffffffffull;
    }
    auto next_step = [&]() {
        const unsigned code = (unsigned)plan_lo;
        plan_lo = (plan_lo >> 12) | ((plan_hi & 0xfffull) << 48);
        plan_hi >>= 12;
        return make_step(rank ^ (int)(code & 7u), (int)((code >> 3) & 1u), (int)((code >> 4) & 7u), ((code >> 7) & 31u) << 4);
    };

    v8i wf[8][2];
    {
        const uint8_t* gsrc;
        unsigned pitch;
        if (cross) {
            pitch = 64;
            gsrc = (const uint8_t*)par + (size_t)peer * REGION + R_W + g * (KD * 64);
        } else {
            if (AG_W_DELAY) __builtin_amdgcn_s_sleep(AG_W_DELAY);
            pitch = NSH;
            gsrc = pw + (c & 7) * 64;
        }
        gsrc += (size_t)(w * 512 + (lane >> 2)) * pitch + (lane & 3) * 16;
        const int grp = lane >> 4, i = lane & 15;
        const unsigned a0 = (unsigned)(((grp >> 1) * 32 + (i >> 1)) * 64 + (grp & 1) * 16 + (i & 1) * 8);
#pragma unroll
        for (int hf = 0; hf < 2; ++hf) {
            if (cross) AG_POLL(flags + (peer * 16 + g * 8) * 128, base + 16u * (hf + 1));
#pragma unroll
            for (int t = 0; t < 16; ++t)
                __builtin_amdgcn_global_load_lds((glb_void*)(gsrc + (size_t)((hf * 256 + t * 16) * pitch)), (lds_void*)(area + t * 1024), 16, 0, 16);
            wait_vm<0>();
#pragma unroll
            for (int s = 0; s < 4; ++s)
#pragma unroll
                for (int t = 0; t < 2; ++t)
#pragma unroll
                    for (int j = 0; j < 4; ++j) {
                        v2i r = __builtin_amdgcn_ds_read_tr8_b64_v2i32(
                            (__attribute__((address_space(3))) v2i*)(area + a0 + (s * 64 + j * 8) * 64 + t * 32));
                        wf[hf * 4 + s][t][2 * j] = r[0];
                        wf[hf * 4 + s][t][2 * j + 1] = r[1];
                    }
            wait_lgkm0();
            __builtin_amdgcn_sched_barrier(0);
        }
    }
    STAMP(1);
    __amdgpu_buffer_rsrc_t rx = mkrsrc(par, PAR_BYTES);
    const unsigned xo = (unsigned)w * 16384u + lane * 16u;
    const int m_w = lane & 31, h_w = lane >> 5;
    const int m_r = w * 4 + (lane >> 4), c_r = lane & 15;
    const unsigned rd_off = (unsigned)(m_r * 256 + ((c_r ^ (m_r & 7)) << 4));
    __amdgpu_buffer_rsrc_t ro = mkrsrc(cross ? p.out[q] : p.out[0], (unsigned)(NR * MSH * NSH * 4));
    const unsigned oo = (unsigned)((m_r * NSH + (cross ? (rank & 3) * 128 + g * 64 : (c & 7) * 64) + c_r * 4) * 4);
    u32x4 xf[16];

#define AG_MFMA(s, lo, hi) do { \
        v8i xa; \
        xa[0] = (lo)[0]; xa[1] = (lo)[1]; xa[2] = (lo)[2]; xa[3] = (lo)[3]; \
        xa[4] = (hi)[0]; xa[5] = (hi)[1]; xa[6] = (hi)[2]; xa[7] = (hi)[3]; \
        acc0 = __builtin_amdgcn_mfma_scale_f32_32x32x64_f8f6f4(wf[s][0], xa, acc0, 1, 1, 0, 0, 0, 0); \
        acc1 = __builtin_amdgcn_mfma_scale_f32_32x32x64_f8f6f4(wf[s][1], xa, acc1, 1, 1, 0, 0, 0, 0); } while (0)
    auto finish = [&](int par2, unsigned orow, f32x16& acc0, f32x16& acc1) {
        if (__builtin_expect(bad, 0)) {
            asm volatile("" ::: "memory");
#pragma unroll
            for (int i = 0; i < 16; ++i) acc0[i] = acc1[i] = __builtin_nanf("");
        }
        char* slot = area + par2 * 8192 + m_w * 256;
#pragma unroll
        for (int t = 0; t < 4; ++t) {
            f32x4 v0 = {acc0[4 * t], acc0[4 * t + 1], acc0[4 * t + 2], acc0[4 * t + 3]};
            f32x4 v1 = {acc1[4 * t], acc1[4 * t + 1], acc1[4 * t + 2], acc1[4 * t + 3]};
            *(f32x4*)(slot + (((2 * t + h_w) ^ (m_w & 7)) << 4)) = v0;
            *(f32x4*)(slot + 128 + (((2 * t + h_w) ^ (m_w & 7)) << 4)) = v1;
        }
        wait_lgkm0();
        __builtin_amdgcn_s_barrier();
        asm volatile("" ::: "memory");
        f32x4 part[8];
#pragma unroll
        for (int v = 0; v < 8; ++v) part[v] = *(const f32x4*)(lds + v * 16384 + par2 * 8192 + rd_off);
        const f32x4 sum = ((part[0] + part[1]) + (part[2] + part[3])) + ((part[4] + part[5]) + (part[6] + part[7]));
        f32x4 o;
#pragma unroll
        for (int i = 0; i < 4; ++i) {
            const float y = sum[i] * scale;
            const float yc = fminf(fmaxf(y, -60.0f), 60.0f);
            o[i] = y * __builtin_amdgcn_rcpf(1.0f + __expf(-yc));
        }
        __builtin_amdgcn_raw_buffer_store_b128(__builtin_bit_cast(u32x4, o), ro, oo + orow, 0, 17);
    };

    u32x4 rx4;
    rx4[0] = __builtin_amdgcn_readfirstlane((unsigned)(uintptr_t)par);
    rx4[1] = __builtin_amdgcn_readfirstlane((unsigned)((uintptr_t)par >> 32)) & 0xffffu;
    rx4[2] = PAR_BYTES;
    rx4[3] = 0x00020000u;
#pragma unroll
    for (int n = 0; n < 16; ++n) xf[n] = u32x4{0u, 0u, 0u, 0u};
    const unsigned vzero = 0;
    Step cur = next_step();
    bool have = false;
#define AG_XLOAD(n) asm volatile("buffer_load_dwordx4 %0, %1, %2, %3 offen offset:%4 sc1" \
        : "+v"(xf[n]) : "v"(xo), "s"(rx4), "s"(cur.xoff + ((n) >> 2) * 4096u), "i"(((n) & 3) * 1024) : "memory")
#define AG_STREAM(s) do { \
        asm volatile("s_waitcnt vmcnt(16)" : "+v"(xf[2 * (s)]), "+v"(xf[2 * (s) + 1])); \
        AG_MFMA(s, xf[2 * (s)], xf[2 * (s) + 1]); \
        __builtin_amdgcn_sched_barrier(0); \
        AG_XLOAD(2 * (s)); AG_XLOAD(2 * (s) + 1); } while (0)
#pragma unroll 1
    for (int i = 0; i < nstep; ++i) {
        if (!have) {
            AG_POLL(flags + cur.foff, base + cur.need);
            AG_XLOAD(0); AG_XLOAD(1); AG_XLOAD(2); AG_XLOAD(3); AG_XLOAD(4); AG_XLOAD(5); AG_XLOAD(6); AG_XLOAD(7);
            AG_XLOAD(8); AG_XLOAD(9); AG_XLOAD(10); AG_XLOAD(11); AG_XLOAD(12); AG_XLOAD(13); AG_XLOAD(14); AG_XLOAD(15);
            __hip_atomic_store(ctr + 16 + w, 0u, __ATOMIC_RELAXED, __HIP_MEMORY_SCOPE_AGENT);
        }
        STAMP(2 + (i & 7) * 3);
        const unsigned orow = cur.orow;
        cur = next_step();
        unsigned fn;
        asm volatile("global_load_dword %0, %1, %2 sc0 sc1" : "=v"(fn) : "v"(vzero), "s"(flags + cur.foff) : "memory");
        f32x16 acc0 = {}, acc1 = {};
        AG_STREAM(0); AG_STREAM(1); AG_STREAM(2); AG_STREAM(3); AG_STREAM(4); AG_STREAM(5); AG_STREAM(6); AG_STREAM(7);
        asm volatile("s_waitcnt vmcnt(16)" : "+v"(fn));
        have = (int)(__builtin_amdgcn_readfirstlane(fn) - (base + cur.need)) >= 0;
        __builtin_amdgcn_sched_barrier(0);
        finish(i & 1, orow, acc0, acc1);
        STAMP(4 + (i & 7) * 3);
    }
    wait_vm<0>();
#pragma unroll
    for (int n = 0; n < 16; ++n) asm volatile("" : "+v"(xf[n]));
#undef AG_STREAM
#undef AG_XLOAD
    if (cross) {
        __builtin_amdgcn_s_barrier();
        if (threadIdx.x == 0 && !bad)
            __hip_atomic_store((unsigned*)(wsp + DONE_OFF) + rank * 32 + c, call, __ATOMIC_RELAXED, __HIP_MEMORY_SCOPE_SYSTEM);
        STAMP(26);
    } else if (q != 0) {
        const Step st = make_step(peer, g, 7, 106u);
        const unsigned lbase = (unsigned)peer * REGION + R_LL + (unsigned)((g * 8 + w) * 8) * 1024u;
        const unsigned lo = lbase + lane * 16u, eo = lbase + 6144u + lane * 8u;
        const unsigned tagw = call << 24;
        u32x4 ll[6];
        u32x2 le;
        AG_POLL(flags + st.foff, base + st.need);
        STAMP(23);
#pragma unroll
        for (int n = 0; n < 10; ++n)
            xf[n] = __builtin_amdgcn_raw_buffer_load_b128(rx, xo + st.xoff + n * 1024, 0, 16);
        le = __builtin_amdgcn_raw_buffer_load_b64(rx, eo, 0, 17);
#pragma unroll
        for (int t = 0; t < 6; ++t) ll[t] = __builtin_amdgcn_raw_buffer_load_b128(rx, lo + t * 1024, 0, 17);
        __builtin_amdgcn_sched_barrier(0);
        f32x16 acc0 = {}, acc1 = {};
#pragma unroll
        for (int s = 0; s < 5; ++s) AG_MFMA(s, xf[2 * s], xf[2 * s + 1]);
        __builtin_amdgcn_sched_barrier(0);
        asm volatile("" : "+v"(acc0), "+v"(acc1));
#define AG_LL_STEP(s) do { \
            for (unsigned spin = 1;; ++spin) { \
                unsigned df = (ll[2 * (s)][3] ^ tagw) | (ll[2 * (s) + 1][3] ^ tagw); \
                if ((s) == 0) df |= le[1] ^ tagw; \
                if (__all((df >> 24) == 0u) || bad) break; \
                if ((spin & 63u) == 0u && __builtin_amdgcn_s_memrealtime() - t_start > DEADLINE_TICKS) bad = true; \
                if ((s) == 0) le = __builtin_amdgcn_raw_buffer_load_b64(rx, eo, 0, 17); \
                _Pragma("unroll") for (int t = 2 * (s); t < 6; ++t) ll[t] = __builtin_amdgcn_raw_buffer_load_b128(rx, lo + t * 1024, 0, 17); \
            } \
            u32x4 a = ll[2 * (s)], cc = ll[2 * (s) + 1]; \
            a[3] = (a[3] & 0x00ffffffu) | ((le[(s) >> 1] >> (16 * ((s) & 1))) << 24); \
            cc[3] = (cc[3] & 0x00ffffffu) | ((le[(s) >> 1] >> (16 * ((s) & 1) + 8)) << 24); \
            if ((s) == 2) STAMP(25); \
            AG_MFMA(5 + (s), a, cc); \
            asm volatile("" : "+v"(acc0), "+v"(acc1)); } while (0)
        AG_LL_STEP(0);
        AG_LL_STEP(1);
        AG_LL_STEP(2);
#undef AG_LL_STEP
        STAMP(24);
        finish(0, st.orow, acc0, acc1);
        STAMP(26);
    } else {
        STAMP(26);
    }
#undef AG_POLL
#undef AG_MFMA
    STAMP_FLUSH();
}

}

extern "C" void kernel_launch(int rank, int nranks, ncclComm_t comm,
                              hipStream_t stream,
                              void* const* d_in, const int* in_sizes, int n_in,
                              void* d_out, int out_size,
                              const int* in_dtypes, int out_dtype,
                              void* const* d_out_all,
                              void* const* d_ws_all, size_t ws_bytes) {
    (void)comm; (void)in_dtypes; (void)out_dtype; (void)d_out;
    if (nranks != NR || (unsigned)rank >= (unsigned)NR || n_in < 4 || ws_bytes < WS_NEED ||
        in_sizes[0] != MSH * KD || in_sizes[1] != KD * NSH || out_size != NR * MSH * NSH) {
        fprintf(stderr, "kernel_launch: unexpected problem shape\n");
        return;
    }
    Params p;
    p.x = (const uint8_t*)d_in[0];
    p.w = (const uint8_t*)d_in[1];
    p.sx = (const float*)d_in[2];
    p.sw = (const float*)d_in[3];
    for (int r = 0; r < NR; ++r) {
        p.ws[r] = (char*)d_ws_all[rank ^ r];
        p.out[r] = (float*)d_out_all[rank ^ r];
    }
    p.rank = rank;
    hipLaunchKernelGGL(ag_gemm_kernel, dim3(NWG), dim3(NTHREADS), 0, stream, p);
    HIP_CHECK(hipGetLastError());
}
```

```cpp
#include <hip/hip_runtime.h>
#include <rccl/rccl.h>
#include <cstdio>
#include <cstdint>

#define HIP_CHECK(expr) do { hipError_t _e = (expr); if (_e != hipSuccess) { \
    fprintf(stderr, "HIP error %s at %s:%d\n", hipGetErrorString(_e), __FILE__, __LINE__); \
    return; } } while (0)

namespace {

typedef unsigned int u32x4 __attribute__((ext_vector_type(4)));
typedef unsigned int u32x2 __attribute__((ext_vector_type(2)));
typedef int v2i __attribute__((ext_vector_type(2)));
typedef int v8i __attribute__((ext_vector_type(8)));
typedef float f32x4 __attribute__((ext_vector_type(4)));
typedef float f32x16 __attribute__((ext_vector_type(16)));
typedef __attribute__((address_space(3))) unsigned lds_u32;
typedef __attribute__((address_space(3))) void lds_void;
typedef const __attribute__((address_space(1))) void glb_void;

constexpr int NR = 8;
constexpr int KD = 4096;
constexpr int MSH = 512;
constexpr int NSH = 512;
constexpr unsigned SHARD_BYTES = MSH * KD;
constexpr unsigned R_W = SHARD_BYTES;
constexpr unsigned R_LL = R_W + 2 * KD * 64;
constexpr unsigned REGION = 3u << 20;
constexpr unsigned PAR_BYTES = NR * REGION;
constexpr size_t FLAG_OFF = 2ull * PAR_BYTES;
constexpr size_t DONE_OFF = FLAG_OFF + NR * 16 * 128;
constexpr size_t CTR_OFF = DONE_OFF + NR * 128;
constexpr size_t WS_NEED = CTR_OFF + 256 * 128;
static_assert(R_LL + 16 * 8192 <= REGION, "region layout");
constexpr unsigned long long DEADLINE_TICKS = 100000000ull;

constexpr int NWG = 256;
constexpr int NTHREADS = 512;
#define AG_WIN 4
#define AG_W_DELAY 32
#define AG_CS 6
#define AG_CW 2
constexpr int WIN = AG_WIN;
constexpr int LDS_SYNC_OFF = 8 * 16384;
constexpr int LDS_BYTES = LDS_SYNC_OFF + 128;

struct Params {
    const uint8_t* x;
    const uint8_t* w;
    const float* sx;
    const float* sw;
    char* ws[NR];
    float* out[NR];
    int rank;
};

__device__ __forceinline__ __amdgpu_buffer_rsrc_t mkrsrc(const void* p, unsigned bytes) {
    unsigned long long a = (unsigned long long)p;
    unsigned lo = __builtin_amdgcn_readfirstlane((unsigned)a);
    unsigned hi = __builtin_amdgcn_readfirstlane((unsigned)(a >> 32));
    return __builtin_amdgcn_make_buffer_rsrc((void*)(((unsigned long long)hi << 32) | lo), 0, bytes, 0x00020000);
}
template <int N> __device__ __forceinline__ void wait_vm() { asm volatile("s_waitcnt vmcnt(%0)" :: "i"(N) : "memory"); }
__device__ __forceinline__ void wait_lgkm0() { asm volatile("s_waitcnt lgkmcnt(0)" ::: "memory"); }
__device__ __forceinline__ unsigned lds_load(lds_u32* p) {
    return __builtin_amdgcn_readfirstlane(__hip_atomic_load(p, __ATOMIC_RELAXED, __HIP_MEMORY_SCOPE_WORKGROUP));
}
__device__ __forceinline__ void lds_add1_asm(lds_u32* p) {
    asm volatile("ds_add_u32 %0, %1" :: "v"((unsigned)(uintptr_t)p), "v"(1u) : "memory");
}
__device__ __forceinline__ void lds_store_asm(lds_u32* p, unsigned v) {
    asm volatile("ds_write_b32 %0, %1" :: "v"((unsigned)(uintptr_t)p), "v"(v) : "memory");
}

#define STAMP(i) do { } while (0)
#define STAMP_FLUSH() do { } while (0)

struct Step { unsigned xoff, foff, need, orow; };
__device__ __forceinline__ Step make_step(int src, int g, int j, unsigned need) {
    Step s;
    s.xoff = (unsigned)src * REGION + (unsigned)g * (1u << 20) + (unsigned)j * (1u << 17);
    s.foff = (unsigned)(src * 16 + g * 8) * 128u;
    s.need = need;
    s.orow = (unsigned)(src * MSH + g * 256 + j * 32) * (NSH * 4u);
    return s;
}
__device__ __forceinline__ unsigned pack_step(int rel, int g, int j, int need) { return (unsigned)(rel | g << 3 | j << 4 | (need >> 4) << 7); }
__device__ __forceinline__ unsigned plan_step(int q, int c, int i) {
    if (q < 4) {
        const int g = c >> 3;
        const int i6 = q == 0 ? 8 : 2, i7 = q == 0 ? 9 : 6;
        if (i == i6) return pack_step(4 + q, g, 6, 48);
        if (i == i7) return pack_step(4 + q, g, 7, 64);
        const int j = i - (i > i6) - (i > i7);
        return pack_step(q, g, j, (j < 4 || q == 0) ? 32 * (j / 2 + 1) : 16 * (j + 1));
    }
    const int e = (c & 7) + 8 * i;
    if (e < 12) return pack_step(0, e & 1, e >> 1, 32 * ((e >> 2) + 1));
    if (e < 24) { const int f = e - 12; return pack_step(1 + (f >> 2), f & 1, (f >> 1) & 1, 32); }
    if (e < 30) { const int f = e - 24; return pack_step(4 + ((q + 1 + (f >> 1)) & 3), f & 1, 6, 48); }
    if (e < 42) { const int f = e - 30; return pack_step(1 + (f >> 2), f & 1, 2 + ((f >> 1) & 1), 64); }
    const int f = e < 48 ? e - 42 : e - 48, j = e < 48 ? 4 : 5;
    return pack_step(1 + ((f >> 1) & 3), f & 1, j, 16 * (j + 1));
}

__global__ void __launch_bounds__(NTHREADS, 2) ag_gemm_kernel(Params p) {
    __shared__ __attribute__((aligned(16))) char lds[LDS_BYTES];
    const int b = blockIdx.x;
    const int lane = threadIdx.x & 63;
    const int wave = __builtin_amdgcn_readfirstlane(threadIdx.x >> 6);
    const int q = b & 7;
    const int idx = b >> 3;
    const int g = (idx >> 3) & 1;
    const uint8_t* const px = p.x;
    const uint8_t* const pw = p.w;
    char* const self = p.ws[0];
    char* const wsp = p.ws[q];
    const int rank = p.rank;
    asm volatile("" :: "s"(px), "s"(pw), "s"(wsp), "s"(self), "s"(rank));
    const int peer = rank ^ q;
    const unsigned long long t_start = __builtin_amdgcn_s_memrealtime();
    lds_u32* sync_words = (lds_u32*)(lds + LDS_SYNC_OFF);
    unsigned* ctr = (unsigned*)(self + CTR_OFF) + b * 32;
    STAMP(0);

    if (__builtin_expect(idx < 16, 1)) {
        const int w = idx & 7;
        const bool cross = q >= 4;
        const int wcol = (peer & 3) * 128 + g * 64;
        if (__builtin_expect(wave == 0, 1)) {
            const int pr = lane & 31, ph = lane >> 5;
            const unsigned prev = __hip_atomic_load(ctr, __ATOMIC_RELAXED, __HIP_MEMORY_SCOPE_AGENT);
            u32x4 va[8], vb[8];
            {
                __amdgpu_buffer_rsrc_t rs = mkrsrc(cross ? pw : px, SHARD_BYTES);
                const unsigned so = cross ? (unsigned)(w * 512 + (lane >> 2)) * NSH + wcol + (lane & 3) * 16
                                          : (unsigned)(g * 256 + pr) * KD + w * 512 + ph * 32;
                const unsigned sa = cross ? 16384u : 64u, sb = cross ? 8192u : 16u;
#pragma unroll
                for (int n = 0; n < 8; ++n) va[n] = __builtin_amdgcn_raw_buffer_load_b128(rs, so, (n >> 1) * sa + (n & 1) * sb, 0);
            }
            if (lane < 8) lds_store_asm(sync_words + lane, 0u);
            const unsigned call = __builtin_amdgcn_readfirstlane(prev) + 1u;
            const unsigned base = call * 256u;
            __amdgpu_buffer_rsrc_t rd = mkrsrc(wsp + (size_t)((call & 1u) * NR + rank) * REGION, REGION);
            unsigned* rflag = (unsigned*)(wsp + FLAG_OFF) + (rank * 16 + idx) * 32;
            const unsigned xs = (unsigned)((g * 64 + w) * 16) * 1024u + lane * 16u;
            const unsigned wsl = R_W + (unsigned)((g * 8 + w) * 32) * 1024u + lane * 16u;
            const int nseg = q == 0 ? 8 : cross ? 4 : 7;
            const unsigned fmask = q == 0 ? 0x54u : cross ? 0x0eu : 0x74u;
            bool bad = false;
            unsigned dprev = cross ? wsl : xs;
            STAMP(1);
#pragma unroll
            for (int n = 0; n < 8; ++n) {
                if (n >= WIN) wait_vm<WIN - 1>();
                __builtin_amdgcn_raw_buffer_store_b128(va[n], rd, dprev + n * 1024, 0, 17);
                if (n == 0) STAMP(3);
                if (n == WIN - 1) {
                    if (lane == 0) lds_store_asm(sync_words + 8, call);
                    wait_lgkm0();
                    __builtin_amdgcn_s_barrier();
                }
            }
#define AG_STAGED(j) do { \
                for (unsigned spin = 1; lds_load(sync_words + (j)) < 4u; ++spin) \
                    if ((spin & 1023u) == 0u && __builtin_amdgcn_s_memrealtime() - t_start > DEADLINE_TICKS) { bad = true; break; } \
                asm volatile("" ::: "memory"); } while (0)
            unsigned ua[8];
#pragma unroll
            for (int n = 0; n < 8; ++n) ua[n] = (unsigned)(pr * 512 + ((((n >> 1) * 4 + ph * 2 + (n & 1)) ^ (pr & 15)) << 4));
            AG_STAGED(0);
#pragma unroll
            for (int n = 0; n < 8; ++n) vb[n] = *(const u32x4*)(lds + 256 + ua[n]);
#pragma unroll 1
            for (int s = 1; s < nseg; ++s) {
                const int jt = cross ? s + 4 : (q != 0 && s == 6) ? 7 : s;
                const unsigned dcur = (cross && s == 1) ? wsl + 16384u : xs + (unsigned)jt * (1u << 17);
                const bool slow_lo = cross && 2 * s >= AG_CS, slow_hi = cross && 2 * s - 1 >= AG_CS;
                AG_STAGED(s);
#pragma unroll
                for (int n = 0; n < 8; ++n) va[n] = *(const u32x4*)(lds + s * 16384 + ua[n]);
#pragma unroll
                for (int n = 0; n < 8; ++n) {
                    if (slow_hi) wait_vm<AG_CW - 1>();
                    wait_vm<WIN - 1>();
                    __builtin_amdgcn_raw_buffer_store_b128(vb[n], rd, dprev + (8 + n) * 1024, 0, 17);
                }
                STAMP(3 + s);
#pragma unroll
                for (int n = 0; n < 8; ++n) vb[n] = *(const u32x4*)(lds + s * 16384 + 256 + ua[n]);
#pragma unroll
                for (int n = 0; n < 8; ++n) {
                    if (slow_lo) wait_vm<AG_CW - 1>();
                    wait_vm<WIN - 1>();
                    if (n == WIN - 1 && ((fmask >> s) & 1u) && !bad) {
                        if (lane == 0) __hip_atomic_store(rflag, base + (unsigned)(s * 16), __ATOMIC_RELAXED, __HIP_MEMORY_SCOPE_SYSTEM);
                    }
                    __builtin_amdgcn_raw_buffer_store_b128(va[n], rd, dcur + n * 1024, 0, 17);
                }
                dprev = dcur;
            }
#pragma unroll
            for (int n = 0; n < 2; ++n) {
                if (cross) wait_vm<AG_CW - 1>();
                wait_vm<WIN - 1>();
                __builtin_amdgcn_raw_buffer_store_b128(vb[n], rd, dprev + (8 + n) * 1024, 0, 17);
            }
            if (q != 0 && !cross) {
                const unsigned tagw = (bad ? call + 1u : call) << 24;
                u32x2 e;
                e[0] = (vb[2][3] >> 24) | ((vb[3][3] >> 24) << 8) | ((vb[4][3] >> 24) << 16) | (vb[5][3] & 0xff000000u);
                e[1] = (vb[6][3] >> 24) | ((vb[7][3] >> 24) << 8) | tagw;
                wait_vm<WIN - 1>();
                __builtin_amdgcn_raw_buffer_store_b64(e, rd, R_LL + (unsigned)(idx * 8 + 6) * 1024u + lane * 8u, 0, 17);
#pragma unroll
                for (int t = 0; t < 6; ++t) {
                    wait_vm<WIN - 1>();
                    if (t == WIN - 2 && !bad) {
                        if (lane == 0) __hip_atomic_store(rflag, base + 106u, __ATOMIC_RELAXED, __HIP_MEMORY_SCOPE_SYSTEM);
                    }
                    u32x4 u = vb[2 + t];
                    u[3] = (u[3] & 0x00ffffffu) | tagw;
                    __builtin_amdgcn_raw_buffer_store_b128(u, rd, R_LL + (unsigned)(idx * 8 + t) * 1024u + lane * 16u, 0, 17);
                }
            } else {
#pragma unroll
                for (int n = 2; n < 8; ++n) {
                    if (cross) wait_vm<AG_CW - 1>();
                    wait_vm<WIN - 1>();
                    __builtin_amdgcn_raw_buffer_store_b128(vb[n], rd, dprev + (8 + n) * 1024, 0, 17);
                }
                wait_vm<0>();
                if (lane == 0 && !bad) __hip_atomic_store(rflag, base + (unsigned)(nseg * 16), __ATOMIC_RELAXED, __HIP_MEMORY_SCOPE_SYSTEM);
            }
#undef AG_STAGED
            if (lane == 0) __hip_atomic_store(ctr, call, __ATOMIC_RELAXED, __HIP_MEMORY_SCOPE_AGENT);
            STAMP(2);
        } else if (wave <= 4) {
            const int v4 = wave - 1;
            const uint8_t* gsrc[4];
            const uint8_t* wsrc[4];
#pragma unroll
            for (int u = 0; u < 4; ++u) {
                const int r = v4 * 8 + u * 2 + (lane >> 5);
                const int ch = (lane & 31) ^ (r & 15);
                gsrc[u] = px + (size_t)(g * 256 + r) * KD + w * 512 + (ch << 4);
                const int un = (ch >> 4) * 8 + ((ch >> 2) & 3) * 2 + (ch & 1), l = r + 32 * ((ch >> 1) & 1);
                wsrc[u] = pw + (size_t)(w * 512 + un * 16 + (l >> 2)) * NSH + wcol + (l & 3) * 16;
            }
#define AG_STAGE(src, off, slot) do { _Pragma("unroll") for (int u = 0; u < 4; ++u) \
                    __builtin_amdgcn_global_load_lds((glb_void*)(src[u] + (size_t)(off)), \
                                                     (lds_void*)(lds + ((slot) * 16 + v4 * 4 + u) * 1024), 16, 0, 0); } while (0)
#define AG_STAGE_X(j, slot) AG_STAGE(gsrc, (j) * 32 * KD, slot)
#define AG_LANDED(slot, n) do { wait_vm<n>(); if (lane == 0) lds_add1_asm(sync_words + (slot)); } while (0)
            if (cross) {
                AG_STAGE(wsrc, 0, 0);
                __builtin_amdgcn_s_barrier();
                AG_LANDED(0, 0);
                AG_STAGE(wsrc, 256 * NSH, 1);
                AG_LANDED(1, 0);
                __builtin_amdgcn_s_sleep(64);
                AG_STAGE_X(6, 2); AG_STAGE_X(7, 3);
                AG_LANDED(2, 4); AG_LANDED(3, 0);
            } else {
                AG_STAGE_X(0, 0);
                __builtin_amdgcn_s_barrier();
                AG_LANDED(0, 0);
                AG_STAGE_X(1, 1);
                AG_LANDED(1, 0);
                __builtin_amdgcn_s_sleep(64);
                AG_STAGE_X(2, 2); AG_STAGE_X(3, 3);
                AG_LANDED(2, 4); AG_LANDED(3, 0);
                if (q != 0) __builtin_amdgcn_s_sleep(127);
                AG_STAGE_X(4, 4); AG_STAGE_X(5, 5);
                AG_LANDED(4, 4); AG_LANDED(5, 0);
                if (q != 0) {
                    __builtin_amdgcn_s_sleep(127);
                    AG_STAGE_X(7, 6);
                    AG_LANDED(6, 0);
                } else {
                    AG_STAGE_X(6, 6); AG_STAGE_X(7, 7);
                    AG_LANDED(6, 4); AG_LANDED(7, 0);
                }
            }
#undef AG_STAGE
#undef AG_STAGE_X
#undef AG_LANDED
            STAMP(1);
        } else {
            __builtin_amdgcn_s_barrier();
            if (b == 0 && wave == 7) {
                asm volatile("" ::: "memory");
                const unsigned call = lds_load(sync_words + 8);
                const unsigned* done = (const unsigned*)(self + DONE_OFF) + (rank ^ (4 + (lane >> 4))) * 32 + (lane & 15);
                for (unsigned spin = 1;; ++spin) {
                    const unsigned f = __hip_atomic_load(done, __ATOMIC_RELAXED, __HIP_MEMORY_SCOPE_SYSTEM);
                    if (__all((int)(f - call) >= 0)) break;
                    if ((spin & 63u) == 0u && __builtin_amdgcn_s_memrealtime() - t_start > DEADLINE_TICKS) {
                        p.out[0][(size_t)rank * MSH * NSH + lane] = __builtin_nanf("");
                        break;
                    }
                    __builtin_amdgcn_s_sleep(1);
                }
                STAMP(2);
            }
        }
        STAMP_FLUSH();
        return;
    }

    const int w = wave;
    const int c = idx - 16;
    const bool cross = q >= 4;
    char* area = lds + w * 16384;
    if (wave == 0) {
        const unsigned prev = __hip_atomic_load(ctr, __ATOMIC_RELAXED, __HIP_MEMORY_SCOPE_AGENT);
        if (lane == 0) {
            __hip_atomic_store(sync_words, prev + 1u, __ATOMIC_RELAXED, __HIP_MEMORY_SCOPE_WORKGROUP);
            __hip_atomic_store(ctr, prev + 1u, __ATOMIC_RELAXED, __HIP_MEMORY_SCOPE_AGENT);
        }
        wait_lgkm0();
    }
    const float scale = p.sx[0] * p.sw[0];
    __builtin_amdgcn_s_barrier();
    asm volatile("" ::: "memory");
    const unsigned call = lds_load(sync_words);
    const unsigned base = call * 256u;
    const char* const par = self + (size_t)(call & 1u) * PAR_BYTES;
    const char* const flags = self + FLAG_OFF + w * 128;
    bool bad = false;

#define AG_POLL(fptr, want) do { \
        for (unsigned spin = 1; !bad; ++spin) { \
            const unsigned f = __builtin_amdgcn_readfirstlane(__hip_atomic_load((const unsigned*)(fptr), __ATOMIC_RELAXED, __HIP_MEMORY_SCOPE_SYSTEM)); \
            if ((int)(f - (want)) >= 0) break; \
            if ((spin & 63u) == 0u && __builtin_amdgcn_s_memrealtime() - t_start > DEADLINE_TICKS) bad = true; \
            __builtin_amdgcn_s_sleep(2); \
        } \
        asm volatile("" ::: "memory"); } while (0)

    const int nstep = q == 0 ? 10 : !cross ? 8 : (c & 7) < 6 ? 7 : 6;
    unsigned long long plan_lo = 0, plan_hi = 0;
#pragma unroll 1
    for (int i = 9; i >= 0; --i) {
        plan_hi = (plan_hi << 12) | (plan_lo >> 48);
        plan_lo = ((plan_lo << 12) | plan_step(q, c, i < nstep ? i : 0)) & 0xfffffffffffffffull;
    }
    auto next_step = [&]() {
        const unsigned code = (unsigned)plan_lo;
        plan_lo = (plan_lo >> 12) | ((plan_hi & 0xfffull) << 48);
        plan_hi >>= 12;
        return make_step(rank ^ (int)(code & 7u), (int)((code >> 3) & 1u), (int)((code >> 4) & 7u), ((code >> 7) & 31u) << 4);
    };

    v8i wf[8][2];
    {
        const uint8_t* gsrc;
        unsigned pitch;
        if (cross) {
            pitch = 64;
            gsrc = (const uint8_t*)par + (size_t)peer * REGION + R_W + g * (KD * 64);
        } else {
            if (AG_W_DELAY) __builtin_amdgcn_s_sleep(AG_W_DELAY);
            pitch = NSH;
            gsrc = pw + (c & 7) * 64;
        }
        gsrc += (size_t)(w * 512 + (lane >> 2)) * pitch + (lane & 3) * 16;
        const int grp = lane >> 4, i = lane & 15;
        const unsigned a0 = (unsigned)(((grp >> 1) * 32 + (i >> 1)) * 64 + (grp & 1) * 16 + (i & 1) * 8);
#pragma unroll
        for (int hf = 0; hf < 2; ++hf) {
            if (cross) AG_POLL(flags + (peer * 16 + g * 8) * 128, base + 16u * (hf + 1));
#pragma unroll
            for (int t = 0; t < 16; ++t)
                __builtin_amdgcn_global_load_lds((glb_void*)(gsrc + (size_t)((hf * 256 + t * 16) * pitch)), (lds_void*)(area + t * 1024), 16, 0, 16);
            wait_vm<0>();
#pragma unroll
            for (int s = 0; s < 4; ++s)
#pragma unroll
                for (int t = 0; t < 2; ++t)
#pragma unroll
                    for (int j = 0; j < 4; ++j) {
                        v2i r = __builtin_amdgcn_ds_read_tr8_b64_v2i32(
                            (__attribute__((address_space(3))) v2i*)(area + a0 + (s * 64 + j * 8) * 64 + t * 32));
                        wf[hf * 4 + s][t][2 * j] = r[0];
                        wf[hf * 4 + s][t][2 * j + 1] = r[1];
                    }
            wait_lgkm0();
            __builtin_amdgcn_sched_barrier(0);
        }
    }
    STAMP(1);
    __amdgpu_buffer_rsrc_t rx = mkrsrc(par, PAR_BYTES);
    const unsigned xo = (unsigned)w * 16384u + lane * 16u;
    const int m_w = lane & 31, h_w = lane >> 5;
    const int m_r = w * 4 + (lane >> 4), c_r = lane & 15;
    const unsigned rd_off = (unsigned)(m_r * 256 + ((c_r ^ (m_r & 7)) << 4));
    __amdgpu_buffer_rsrc_t ro = mkrsrc(cross ? p.out[q] : p.out[0], (unsigned)(NR * MSH * NSH * 4));
    const unsigned oo = (unsigned)((m_r * NSH + (cross ? (rank & 3) * 128 + g * 64 : (c & 7) * 64) + c_r * 4) * 4);
    u32x4 xf[16];

#define AG_MFMA(s, lo, hi) do { \
        v8i xa; \
        xa[0] = (lo)[0]; xa[1] = (lo)[1]; xa[2] = (lo)[2]; xa[3] = (lo)[3]; \
        xa[4] = (hi)[0]; xa[5] = (hi)[1]; xa[6] = (hi)[2]; xa[7] = (hi)[3]; \
        acc0 = __builtin_amdgcn_mfma_scale_f32_32x32x64_f8f6f4(wf[s][0], xa, acc0, 1, 1, 0, 0, 0, 0); \
        acc1 = __builtin_amdgcn_mfma_scale_f32_32x32x64_f8f6f4(wf[s][1], xa, acc1, 1, 1, 0, 0, 0, 0); } while (0)
    auto finish = [&](int par2, unsigned orow, f32x16& acc0, f32x16& acc1) {
        if (__builtin_expect(bad, 0)) {
            asm volatile("" ::: "memory");
#pragma unroll
            for (int i = 0; i < 16; ++i) acc0[i] = acc1[i] = __builtin_nanf("");
        }
        char* slot = area + par2 * 8192 + m_w * 256;
#pragma unroll
        for (int t = 0; t < 4; ++t) {
            f32x4 v0 = {acc0[4 * t], acc0[4 * t + 1], acc0[4 * t + 2], acc0[4 * t + 3]};
            f32x4 v1 = {acc1[4 * t], acc1[4 * t + 1], acc1[4 * t + 2], acc1[4 * t + 3]};
            *(f32x4*)(slot + (((2 * t + h_w) ^ (m_w & 7)) << 4)) = v0;
            *(f32x4*)(slot + 128 + (((2 * t + h_w) ^ (m_w & 7)) << 4)) = v1;
        }
        wait_lgkm0();
        __builtin_amdgcn_s_barrier();
        asm volatile("" ::: "memory");
        f32x4 part[8];
#pragma unroll
        for (int v = 0; v < 8; ++v) part[v] = *(const f32x4*)(lds + v * 16384 + par2 * 8192 + rd_off);
        const f32x4 sum = ((part[0] + part[1]) + (part[2] + part[3])) + ((part[4] + part[5]) + (part[6] + part[7]));
        f32x4 o;
#pragma unroll
        for (int i = 0; i < 4; ++i) {
            const float y = sum[i] * scale;
            const float yc = fminf(fmaxf(y, -60.0f), 60.0f);
            o[i] = y * __builtin_amdgcn_rcpf(1.0f + __expf(-yc));
        }
        __builtin_amdgcn_raw_buffer_store_b128(__builtin_bit_cast(u32x4, o), ro, oo + orow, 0, 17);
    };

    u32x4 rx4;
    rx4[0] = __builtin_amdgcn_readfirstlane((unsigned)(uintptr_t)par);
    rx4[1] = __builtin_amdgcn_readfirstlane((unsigned)((uintptr_t)par >> 32)) & 0xffffu;
    rx4[2] = PAR_BYTES;
    rx4[3] = 0x00020000u;
#pragma unroll
    for (int n = 0; n < 16; ++n) xf[n] = u32x4{0u, 0u, 0u, 0u};
    const unsigned vzero = 0;
    Step cur = next_step();
    bool have = false;
#define AG_XLOAD(n) asm volatile("buffer_load_dwordx4 %0, %1, %2, %3 offen offset:%4 sc1" \
        : "+v"(xf[n]) : "v"(xo), "s"(rx4), "s"(cur.xoff + ((n) >> 2) * 4096u), "i"(((n) & 3) * 1024) : "memory")
#define AG_STREAM(s) do { \
        asm volatile("s_waitcnt vmcnt(16)" : "+v"(xf[2 * (s)]), "+v"(xf[2 * (s) + 1])); \
        AG_MFMA(s, xf[2 * (s)], xf[2 * (s) + 1]); \
        __builtin_amdgcn_sched_barrier(0); \
        AG_XLOAD(2 * (s)); AG_XLOAD(2 * (s) + 1); } while (0)
#pragma unroll 1
    for (int i = 0; i < nstep; ++i) {
        if (!have) {
            AG_POLL(flags + cur.foff, base + cur.need);
            AG_XLOAD(0); AG_XLOAD(1); AG_XLOAD(2); AG_XLOAD(3); AG_XLOAD(4); AG_XLOAD(5); AG_XLOAD(6); AG_XLOAD(7);
            AG_XLOAD(8); AG_XLOAD(9); AG_XLOAD(10); AG_XLOAD(11); AG_XLOAD(12); AG_XLOAD(13); AG_XLOAD(14); AG_XLOAD(15);
            __hip_atomic_store(ctr + 16 + w, 0u, __ATOMIC_RELAXED, __HIP_MEMORY_SCOPE_AGENT);
        }
        STAMP(2 + (i & 7) * 3);
        const unsigned orow = cur.orow;
        cur = next_step();
        unsigned fn;
        asm volatile("global_load_dword %0, %1, %2 sc0 sc1" : "=v"(fn) : "v"(vzero), "s"(flags + cur.foff) : "memory");
        f32x16 acc0 = {}, acc1 = {};
        AG_STREAM(0); AG_STREAM(1); AG_STREAM(2); AG_STREAM(3); AG_STREAM(4); AG_STREAM(5); AG_STREAM(6); AG_STREAM(7);
        asm volatile("s_waitcnt vmcnt(16)" : "+v"(fn));
        have = (int)(__builtin_amdgcn_readfirstlane(fn) - (base + cur.need)) >= 0;
        __builtin_amdgcn_sched_barrier(0);
        finish(i & 1, orow, acc0, acc1);
        STAMP(4 + (i & 7) * 3);
    }
    wait_vm<0>();
#pragma unroll
    for (int n = 0; n < 16; ++n) asm volatile("" : "+v"(xf[n]));
#undef AG_STREAM
#undef AG_XLOAD
    if (cross) {
        __builtin_amdgcn_s_barrier();
        if (threadIdx.x == 0 && !bad)
            __hip_atomic_store((unsigned*)(wsp + DONE_OFF) + rank * 32 + c, call, __ATOMIC_RELAXED, __HIP_MEMORY_SCOPE_SYSTEM);
        STAMP(26);
    } else if (q != 0) {
        const Step st = make_step(peer, g, 7, 106u);
        const unsigned lbase = (unsigned)peer * REGION + R_LL + (unsigned)((g * 8 + w) * 8) * 1024u;
        const unsigned lo = lbase + lane * 16u, eo = lbase + 6144u + lane * 8u;
        const unsigned tagw = call << 24;
        u32x4 ll[6];
        u32x2 le;
        AG_POLL(flags + st.foff, base + st.need);
        STAMP(23);
#pragma unroll
        for (int n = 0; n < 10; ++n)
            xf[n] = __builtin_amdgcn_raw_buffer_load_b128(rx, xo + st.xoff + n * 1024, 0, 16);
        le = __builtin_amdgcn_raw_buffer_load_b64(rx, eo, 0, 17);
#pragma unroll
        for (int t = 0; t < 6; ++t) ll[t] = __builtin_amdgcn_raw_buffer_load_b128(rx, lo + t * 1024, 0, 17);
        __builtin_amdgcn_sched_barrier(0);
        f32x16 acc0 = {}, acc1 = {};
#pragma unroll
        for (int s = 0; s < 5; ++s) AG_MFMA(s, xf[2 * s], xf[2 * s + 1]);
        __builtin_amdgcn_sched_barrier(0);
        asm volatile("" : "+v"(acc0), "+v"(acc1));
#define AG_LL_STEP(s) do { \
            for (unsigned spin = 1;; ++spin) { \
                unsigned df = (ll[2 * (s)][3] ^ tagw) | (ll[2 * (s) + 1][3] ^ tagw); \
                if ((s) == 0) df |= le[1] ^ tagw; \
                if (__all((df >> 24) == 0u) || bad) break; \
                if ((spin & 63u) == 0u && __builtin_amdgcn_s_memrealtime() - t_start > DEADLINE_TICKS) bad = true; \
                if ((s) == 0) le = __builtin_amdgcn_raw_buffer_load_b64(rx, eo, 0, 17); \
                _Pragma("unroll") for (int t = 2 * (s); t < 6; ++t) ll[t] = __builtin_amdgcn_raw_buffer_load_b128(rx, lo + t * 1024, 0, 17); \
            } \
            u32x4 a = ll[2 * (s)], cc = ll[2 * (s) + 1]; \
            a[3] = (a[3] & 0x00ffffffu) | ((le[(s) >> 1] >> (16 * ((s) & 1))) << 24); \
            cc[3] = (cc[3] & 0x00ffffffu) | ((le[(s) >> 1] >> (16 * ((s) & 1) + 8)) << 24); \
            if ((s) == 2) STAMP(25); \
            AG_MFMA(5 + (s), a, cc); \
            asm volatile("" : "+v"(acc0), "+v"(acc1)); } while (0)
        AG_LL_STEP(0);
        AG_LL_STEP(1);
        AG_LL_STEP(2);
#undef AG_LL_STEP
        STAMP(24);
        finish(0, st.orow, acc0, acc1);
        STAMP(26);
    } else {
        STAMP(26);
    }
#undef AG_POLL
#undef AG_MFMA
    STAMP_FLUSH();
}

}

extern "C" void kernel_launch(int rank, int nranks, ncclComm_t comm,
                              hipStream_t stream,
                              void* const* d_in, const int* in_sizes, int n_in,
                              void* d_out, int out_size,
                              const int* in_dtypes, int out_dtype,
                              void* const* d_out_all,
                              void* const* d_ws_all, size_t ws_bytes) {
    (void)comm; (void)in_dtypes; (void)out_dtype; (void)d_out;
    if (nranks != NR || (unsigned)rank >= (unsigned)NR || n_in < 4 || ws_bytes < WS_NEED ||
        in_sizes[0] != MSH * KD || in_sizes[1] != KD * NSH || out_size != NR * MSH * NSH) {
        fprintf(stderr, "kernel_launch: unexpected problem shape\n");
        return;
    }
    Params p;
    p.x = (const uint8_t*)d_in[0];
    p.w = (const uint8_t*)d_in[1];
    p.sx = (const float*)d_in[2];
    p.sw = (const float*)d_in[3];
    for (int r = 0; r < NR; ++r) {
        p.ws[r] = (char*)d_ws_all[rank ^ r];
        p.out[r] = (float*)d_out_all[rank ^ r];
    }
    p.rank = rank;
    hipLaunchKernelGGL(ag_gemm_kernel, dim3(NWG), dim3(NTHREADS), 0, stream, p);
    HIP_CHECK(hipGetLastError());
}
```

```cpp
#include <hip/hip_runtime.h>
#include <rccl/rccl.h>
#include <cstdio>
#include <cstdint>

#define HIP_CHECK(expr) do { hipError_t _e = (expr); if (_e != hipSuccess) { \
    fprintf(stderr, "HIP error %s at %s:%d\n", hipGetErrorString(_e), __FILE__, __LINE__); \
    return; } } while (0)

namespace {

typedef unsigned int u32x4 __attribute__((ext_vector_type(4)));
typedef unsigned int u32x2 __attribute__((ext_vector_type(2)));
typedef int v2i __attribute__((ext_vector_type(2)));
typedef int v8i __attribute__((ext_vector_type(8)));
typedef float f32x4 __attribute__((ext_vector_type(4)));
typedef float f32x16 __attribute__((ext_vector_type(16)));
typedef __attribute__((address_space(3))) unsigned lds_u32;
typedef __attribute__((address_space(3))) void lds_void;
typedef const __attribute__((address_space(1))) void glb_void;

constexpr int NR = 8;
constexpr int KD = 4096;
constexpr int MSH = 512;
constexpr int NSH = 512;
constexpr unsigned SHARD_BYTES = MSH * KD;
constexpr unsigned R_W = SHARD_BYTES;
constexpr unsigned R_LL = R_W + 2 * KD * 64;
constexpr unsigned REGION = 3u << 20;
constexpr unsigned PAR_BYTES = NR * REGION;
constexpr size_t FLAG_OFF = 2ull * PAR_BYTES;
constexpr size_t DONE_OFF = FLAG_OFF + NR * 16 * 128;
constexpr size_t CTR_OFF = DONE_OFF + NR * 128;
constexpr size_t ROLE_OFF = CTR_OFF + 256 * 128;
constexpr size_t WS_NEED = ROLE_OFF + 256;
static_assert(R_LL + 16 * 8192 <= REGION, "region layout");
constexpr unsigned DEADLINE_TICKS = 100000000u;

constexpr int NWG = 256;
constexpr int NTHREADS = 512;
#define AG_WIN 4
#define AG_W_DELAY 32
#define AG_I6 4
#define AG_OFF 2
#define AG_XS 2
#define AG_XW 6
constexpr int WIN = AG_WIN;
constexpr int LDS_SYNC_OFF = 8 * 16384;
constexpr int LDS_BYTES = LDS_SYNC_OFF + 128;

struct Params {
    const uint8_t* x;
    const uint8_t* w;
    const float* sx;
    const float* sw;
    char* ws[NR];
    float* out[NR];
    int rank;
};

__device__ __forceinline__ __amdgpu_buffer_rsrc_t mkrsrc(const void* p, unsigned bytes) {
    unsigned long long a = (unsigned long long)p;
    unsigned lo = __builtin_amdgcn_readfirstlane((unsigned)a);
    unsigned hi = __builtin_amdgcn_readfirstlane((unsigned)(a >> 32));
    return __builtin_amdgcn_make_buffer_rsrc((void*)(((unsigned long long)hi << 32) | lo), 0, bytes, 0x00020000);
}
template <int N> __device__ __forceinline__ void wait_vm() { asm volatile("s_waitcnt vmcnt(%0)" :: "i"(N) : "memory"); }
__device__ __forceinline__ void wait_lgkm0() { asm volatile("s_waitcnt lgkmcnt(0)" ::: "memory"); }
__device__ __forceinline__ unsigned lds_load(lds_u32* p) {
    return __builtin_amdgcn_readfirstlane(__hip_atomic_load(p, __ATOMIC_RELAXED, __HIP_MEMORY_SCOPE_WORKGROUP));
}
__device__ __forceinline__ unsigned lds_load_asm(lds_u32* p) {
    unsigned v;
    asm volatile("ds_read_b32 %0, %1\n\ts_waitcnt lgkmcnt(0)" : "=v"(v) : "v"((unsigned)(uintptr_t)p) : "memory");
    return __builtin_amdgcn_readfirstlane(v);
}
__device__ __forceinline__ void lds_add1_asm(lds_u32* p) {
    asm volatile("ds_add_u32 %0, %1" :: "v"((unsigned)(uintptr_t)p), "v"(1u) : "memory");
}
__device__ __forceinline__ void lds_put_asm(lds_u32* p, unsigned v) {
    asm volatile("ds_write_b32 %0, %1" :: "v"((unsigned)(uintptr_t)p), "v"(v) : "memory");
}

#define STAMP(i) do { } while (0)
#define STAMP_FLUSH() do { } while (0)

#define AG_REAL 0x45172630u
__host__ __device__ __forceinline__ int real_rank(int v) { return (int)((AG_REAL >> (4 * v)) & 7u); }

struct Step { unsigned xoff, foff, need, orow; };
__device__ __forceinline__ Step make_step(int src, int g, int j, unsigned need) {
    Step s;
    s.xoff = (unsigned)src * REGION + (unsigned)g * (1u << 20) + (unsigned)j * (1u << 17);
    s.foff = (unsigned)(src * 16 + g * 8) * 128u;
    s.need = need;
    s.orow = (unsigned)(real_rank(src) * MSH + g * 256 + j * 32) * (NSH * 4u);
    return s;
}
__device__ __forceinline__ unsigned pack_step(int rel, int g, int j, int need) { return (unsigned)(rel | g << 3 | j << 4 | (need >> 4) << 7); }
__device__ __forceinline__ unsigned plan_step(int q, int c, int i) {
    if (q < 4) {
        const int g = c >> 3;
        if (q == 0) {
            const int k = i + 3 * AG_OFF;
            return k < 8 ? pack_step(0, g, k, 32 * (k / 2 + 1)) : k == 8 ? pack_step(4, g, 6, 48) : pack_step(4 + ((k - 9) & 3), g, 7, 64);
        }
        if (i < AG_OFF) return pack_step(0, g, (q - 1) * AG_OFF + i, 32 * (((q - 1) * AG_OFF + i) / 2 + 1));
        const int k = i - AG_OFF;
        if (k == AG_I6) return pack_step(4 + q, g, 6, 48);
        const int j = k - (k > AG_I6);
        return pack_step(q, g, j, j < 4 ? 32 * (j / 2 + 1) : 16 * (j + 1));
    }
    const int e = (c & 7) + 8 * i;
    if (e < 12) return pack_step(0, e & 1, e >> 1, 32 * ((e >> 2) + 1));
    if (e < 24) { const int f = e - 12; return pack_step(1 + (f >> 2), f & 1, (f >> 1) & 1, 32); }
    if (e < 30) { const int f = e - 24; return pack_step(4 + ((q + 1 + (f >> 1)) & 3), f & 1, 6, 48); }
    if (e < 42) { const int f = e - 30; return pack_step(1 + (f >> 2), f & 1, 2 + ((f >> 1) & 1), 64); }
    const int f = e < 48 ? e - 42 : e - 48, j = e < 48 ? 4 : 5;
    return pack_step(1 + ((f >> 1) & 3), f & 1, j, 16 * (j + 1));
}

#define AG_STAMP_ARG
#define AG_STAMP_PASS

template <bool FIRST>
__device__ __forceinline__ bool run_role(const Params& p, char* lds, const int q, const int idx, const uint8_t* const px,
                                         const uint8_t* const pw, char* const self, char* const wsp, const int rank,
                                         const unsigned t_start, const unsigned call_in AG_STAMP_ARG) {
    const int b = blockIdx.x;
    int tid = threadIdx.x;
    if (!FIRST) asm volatile("" : "+v"(tid));
    const int lane = tid & 63;
    const int wave = __builtin_amdgcn_readfirstlane(tid >> 6);
    const int g = (idx >> 3) & 1;
    const int peer = rank ^ q;
    lds_u32* sync_words = (lds_u32*)(lds + LDS_SYNC_OFF);
    unsigned* ctr = (unsigned*)(self + CTR_OFF) + b * 32;
#define AG_CLAIM_ROLE(cl) do { \
        const unsigned cl_ = (cl); \
        if (lane == 0) { \
            const unsigned old_ = __hip_atomic_exchange((unsigned*)(self + CTR_OFF) + (idx * 8 + q) * 32 + 8, cl_, __ATOMIC_RELAXED, __HIP_MEMORY_SCOPE_AGENT); \
            lds_put_asm(sync_words + 9, old_ == cl_ ? 2u : 1u); \
            __hip_atomic_fetch_add((unsigned*)(self + ROLE_OFF), 1u, __ATOMIC_RELAXED, __HIP_MEMORY_SCOPE_AGENT); \
            __hip_atomic_store(ctr, cl_, __ATOMIC_RELAXED, __HIP_MEMORY_SCOPE_AGENT); \
        } \
        wait_lgkm0(); } while (0)
    auto role_taken = [&]() {
        unsigned v;
        while ((v = lds_load_asm(sync_words + 9)) == 0u) __builtin_amdgcn_s_sleep(1);
        return v == 2u;
    };
#define AG_TAKEN() (FIRST && role_taken())

    if (__builtin_expect(idx < 16, 1)) {
        const int w = idx & 7;
        const bool cross = q >= 4;
        const int wcol = (peer & 3) * 128 + g * 64;
        if (__builtin_expect(wave == 0, 1)) {
            const int pr = lane & 31, ph = lane >> 5;
            const unsigned prev = FIRST ? __hip_atomic_load(ctr, __ATOMIC_RELAXED, __HIP_MEMORY_SCOPE_AGENT) : call_in - 1u;
            u32x4 va[8], vb[8];
            {
                __amdgpu_buffer_rsrc_t rs = mkrsrc(cross ? pw : px, SHARD_BYTES);
                const unsigned so = cross ? (unsigned)(w * 512 + (lane >> 2)) * NSH + wcol + (lane & 3) * 16
                                          : (unsigned)(g * 256 + pr) * KD + w * 512 + ph * 32;
                const unsigned sa = cross ? 16384u : 64u, sb = cross ? 8192u : 16u;
#pragma unroll
                for (int n = 0; n < 8; ++n) va[n] = __builtin_amdgcn_raw_buffer_load_b128(rs, so, (n >> 1) * sa + (n & 1) * sb, 0);
            }
            if (lane < 10) lds_put_asm(sync_words + lane, 0u);
            const unsigned call = __builtin_amdgcn_readfirstlane(prev) + 1u;
            const unsigned base = call * 256u;
            __amdgpu_buffer_rsrc_t rd = mkrsrc(wsp + (size_t)((call & 1u) * NR + rank) * REGION, REGION);
            unsigned* rflag = (unsigned*)(wsp + FLAG_OFF) + (rank * 16 + idx) * 32;
            const unsigned xs = (unsigned)((g * 64 + w) * 16) * 1024u + lane * 16u;
            const unsigned wsl = R_W + (unsigned)((g * 8 + w) * 32) * 1024u + lane * 16u;
            const int nseg = q == 0 ? 8 : cross ? 4 : 7;
            const unsigned fmask = q == 0 ? 0x54u : cross ? 0x0eu : 0x74u;
            bool bad = false;
            unsigned dprev = cross ? wsl : xs;
            STAMP(1);
#pragma unroll
            for (int n = 0; n < 8; ++n) {
                if (n >= WIN) wait_vm<WIN - 1>();
                __builtin_amdgcn_raw_buffer_store_b128(va[n], rd, dprev + n * 1024, 0, 17);
                if (n == 0) STAMP(3);
                if (n == WIN - 1) {
                    if (lane == 0) lds_put_asm(sync_words + 8, call);
                    wait_lgkm0();
                    __builtin_amdgcn_s_barrier();
                }
            }
#define AG_STAGED(j) do { \
                for (unsigned spin = 1; lds_load(sync_words + (j)) < 4u; ++spin) \
                    if ((spin & 1023u) == 0u && (unsigned)__builtin_amdgcn_s_memrealtime() - t_start > DEADLINE_TICKS) { bad = true; break; } \
                asm volatile("" ::: "memory"); } while (0)
            unsigned ua[8];
#pragma unroll
            for (int n = 0; n < 8; ++n) ua[n] = (unsigned)(pr * 512 + ((((n >> 1) * 4 + ph * 2 + (n & 1)) ^ (pr & 15)) << 4));
            AG_STAGED(0);
#pragma unroll
            for (int n = 0; n < 8; ++n) vb[n] = *(const u32x4*)(lds + 256 + ua[n]);
#pragma unroll 1
            for (int s = 1; s < nseg; ++s) {
                const int jt = cross ? s + 4 : (q != 0 && s == 6) ? 7 : s;
                const unsigned dcur = (cross && s == 1) ? wsl + 16384u : xs + (unsigned)jt * (1u << 17);
                const bool wide_lo = cross && 2 * s >= AG_XS, wide_hi = cross && 2 * s - 1 >= AG_XS;
                AG_STAGED(s);
#pragma unroll
                for (int n = 0; n < 8; ++n) va[n] = *(const u32x4*)(lds + s * 16384 + ua[n]);
#pragma unroll
                for (int n = 0; n < 8; ++n) {
                    if (wide_hi) wait_vm<AG_XW - 1>(); else
                    wait_vm<WIN - 1>();
                    __builtin_amdgcn_raw_buffer_store_b128(vb[n], rd, dprev + (8 + n) * 1024, 0, 17);
                }
                STAMP(3 + s);
#pragma unroll
                for (int n = 0; n < 8; ++n) vb[n] = *(const u32x4*)(lds + s * 16384 + 256 + ua[n]);
#pragma unroll
                for (int n = 0; n < 8; ++n) {
                    if (wide_lo) wait_vm<AG_XW - 1>(); else
                    wait_vm<WIN - 1>();
                    if (n == (wide_lo ? AG_XW - 1 : WIN - 1) && ((fmask >> s) & 1u) && !bad) {
                        if (lane == 0) __hip_atomic_store(rflag, base + (unsigned)(s * 16), __ATOMIC_RELAXED, __HIP_MEMORY_SCOPE_SYSTEM);
                    }
                    __builtin_amdgcn_raw_buffer_store_b128(va[n], rd, dcur + n * 1024, 0, 17);
                }
                dprev = dcur;
            }
#pragma unroll
            for (int n = 0; n < 2; ++n) {
                if (cross) wait_vm<AG_XW - 1>(); else
                wait_vm<WIN - 1>();
                __builtin_amdgcn_raw_buffer_store_b128(vb[n], rd, dprev + (8 + n) * 1024, 0, 17);
            }
            if (q != 0 && !cross) {
                const unsigned tagw = (bad ? call + 1u : call) << 24;
                u32x2 e;
                e[0] = (vb[2][3] >> 24) | ((vb[3][3] >> 24) << 8) | ((vb[4][3] >> 24) << 16) | (vb[5][3] & 0xff000000u);
                e[1] = (vb[6][3] >> 24) | ((vb[7][3] >> 24) << 8) | tagw;
                wait_vm<WIN - 1>();
                __builtin_amdgcn_raw_buffer_store_b64(e, rd, R_LL + (unsigned)(idx * 8 + 6) * 1024u + lane * 8u, 0, 17);
#pragma unroll
                for (int t = 0; t < 6; ++t) {
                    wait_vm<WIN - 1>();
                    if (t == WIN - 2 && !bad) {
                        if (lane == 0) __hip_atomic_store(rflag, base + 106u, __ATOMIC_RELAXED, __HIP_MEMORY_SCOPE_SYSTEM);
                    }
                    u32x4 u = vb[2 + t];
                    u[3] = (u[3] & 0x00ffffffu) | tagw;
                    __builtin_amdgcn_raw_buffer_store_b128(u, rd, R_LL + (unsigned)(idx * 8 + t) * 1024u + lane * 16u, 0, 17);
                }
            } else {
#pragma unroll
                for (int n = 2; n < 8; ++n) {
                    wait_vm<WIN - 1>();
                    __builtin_amdgcn_raw_buffer_store_b128(vb[n], rd, dprev + (8 + n) * 1024, 0, 17);
                }
                wait_vm<0>();
                if (lane == 0 && !bad) __hip_atomic_store(rflag, base + (unsigned)(nseg * 16), __ATOMIC_RELAXED, __HIP_MEMORY_SCOPE_SYSTEM);
            }
#undef AG_STAGED
            STAMP(2);
        } else if (wave <= 4) {
            const int v4 = wave - 1;
            const uint8_t* gsrc[4];
            const uint8_t* wsrc[4];
#pragma unroll
            for (int u = 0; u < 4; ++u) {
                const int r = v4 * 8 + u * 2 + (lane >> 5);
                const int ch = (lane & 31) ^ (r & 15);
                gsrc[u] = px + (size_t)(g * 256 + r) * KD + w * 512 + (ch << 4);
                const int un = (ch >> 4) * 8 + ((ch >> 2) & 3) * 2 + (ch & 1), l = r + 32 * ((ch >> 1) & 1);
                wsrc[u] = pw + (size_t)(w * 512 + un * 16 + (l >> 2)) * NSH + wcol + (l & 3) * 16;
            }
#define AG_STAGE(src, off, slot) do { _Pragma("unroll") for (int u = 0; u < 4; ++u) \
                    __builtin_amdgcn_global_load_lds((glb_void*)(src[u] + (size_t)(off)), \
                                                     (lds_void*)(lds + ((slot) * 16 + v4 * 4 + u) * 1024), 16, 0, 0); } while (0)
#define AG_STAGE_X(j, slot) AG_STAGE(gsrc, (j) * 32 * KD, slot)
#define AG_LANDED(slot, n) do { wait_vm<n>(); if (lane == 0) lds_add1_asm(sync_words + (slot)); } while (0)
            if (cross) {
                AG_STAGE(wsrc, 0, 0);
                __builtin_amdgcn_s_barrier();
                AG_LANDED(0, 0);
                AG_STAGE(wsrc, 256 * NSH, 1);
                AG_LANDED(1, 0);
                __builtin_amdgcn_s_sleep(64);
                AG_STAGE_X(6, 2); AG_STAGE_X(7, 3);
                AG_LANDED(2, 4); AG_LANDED(3, 0);
            } else {
                AG_STAGE_X(0, 0);
                __builtin_amdgcn_s_barrier();
                AG_LANDED(0, 0);
                AG_STAGE_X(1, 1);
                AG_LANDED(1, 0);
                __builtin_amdgcn_s_sleep(64);
                AG_STAGE_X(2, 2); AG_STAGE_X(3, 3);
                AG_LANDED(2, 4); AG_LANDED(3, 0);
                if (q != 0) __builtin_amdgcn_s_sleep(127);
                AG_STAGE_X(4, 4); AG_STAGE_X(5, 5);
                AG_LANDED(4, 4); AG_LANDED(5, 0);
                if (q != 0) {
                    __builtin_amdgcn_s_sleep(127);
                    AG_STAGE_X(7, 6);
                    AG_LANDED(6, 0);
                } else {
                    AG_STAGE_X(6, 6); AG_STAGE_X(7, 7);
                    AG_LANDED(6, 4); AG_LANDED(7, 0);
                }
            }
#undef AG_STAGE
#undef AG_STAGE_X
#undef AG_LANDED
            STAMP(1);
        } else {
            __builtin_amdgcn_s_barrier();
            if (FIRST && wave == 5) AG_CLAIM_ROLE(lds_load_asm(sync_words + 8));
            if (q == 0 && idx == 0 && wave == 7 && !AG_TAKEN()) {
                asm volatile("" ::: "memory");
                const unsigned call = lds_load(sync_words + 8);
                const unsigned* done = (const unsigned*)(self + DONE_OFF) + (rank ^ (4 + (lane >> 4))) * 32 + (lane & 15);
                for (unsigned spin = 1;; ++spin) {
                    const unsigned f = __hip_atomic_load(done, __ATOMIC_RELAXED, __HIP_MEMORY_SCOPE_SYSTEM);
                    if (__all((int)(f - call) >= 0)) break;
                    if ((spin & 63u) == 0u && (unsigned)__builtin_amdgcn_s_memrealtime() - t_start > DEADLINE_TICKS) {
                        p.out[0][(size_t)real_rank(rank) * MSH * NSH + lane] = __builtin_nanf("");
                        break;
                    }
                    __builtin_amdgcn_s_sleep(1);
                }
                STAMP(2);
            }
        }
        STAMP_FLUSH();
        return AG_TAKEN();
    }

    const int w = wave;
    const int c = idx - 16;
    const bool cross = q >= 4;
    char* area = lds + w * 16384;
    if (wave == 0) {
        const unsigned cl = FIRST ? __builtin_amdgcn_readfirstlane(__hip_atomic_load(ctr, __ATOMIC_RELAXED, __HIP_MEMORY_SCOPE_AGENT)) + 1u : call_in;
        if (lane == 0) lds_put_asm(sync_words + 8, cl);
        if (FIRST) AG_CLAIM_ROLE(cl);
        wait_lgkm0();
    }
    const float scale = p.sx[0] * p.sw[0];
    __builtin_amdgcn_s_barrier();
    if (__builtin_expect(AG_TAKEN(), 0)) return true;
    asm volatile("" ::: "memory");
    const unsigned call = lds_load(sync_words + 8);
    const unsigned base = call * 256u;
    const char* const par = self + (size_t)(call & 1u) * PAR_BYTES;
    const char* const flags = self + FLAG_OFF + w * 128;
    bool bad = false;

#define AG_POLL(fptr, want) do { \
        for (unsigned spin = 1; !bad; ++spin) { \
            const unsigned f = __builtin_amdgcn_readfirstlane(__hip_atomic_load((const unsigned*)(fptr), __ATOMIC_RELAXED, __HIP_MEMORY_SCOPE_SYSTEM)); \
            if ((int)(f - (want)) >= 0) break; \
            if ((spin & 63u) == 0u && (unsigned)__builtin_amdgcn_s_memrealtime() - t_start > DEADLINE_TICKS) bad = true; \
            __builtin_amdgcn_s_sleep(2); \
        } \
        asm volatile("" ::: "memory"); } while (0)

    const int nstep = __builtin_amdgcn_readfirstlane(q == 0 ? 13 - 3 * AG_OFF : !cross ? 7 + AG_OFF : (c & 7) < 6 ? 7 : 6);
    unsigned long long plan_lo = 0, plan_mid = 0, plan_hi = 0;
#pragma unroll 1
    for (int i = 12; i >= 0; --i) {
        plan_hi = (plan_hi << 12) | (plan_mid >> 48);
        plan_mid = ((plan_mid << 12) | (plan_lo >> 48)) & 0xfffffffffffffffull;
        plan_lo = ((plan_lo << 12) | plan_step(q, c, i < nstep ? i : 0)) & 0xfffffffffffffffull;
    }
    auto next_step = [&]() {
        const unsigned code = (unsigned)plan_lo;
        plan_lo = (plan_lo >> 12) | ((plan_mid & 0xfffull) << 48);
        plan_mid = (plan_mid >> 12) | ((plan_hi & 0xfffull) << 48);
        plan_hi >>= 12;
        return make_step(rank ^ (int)(code & 7u), (int)((code >> 3) & 1u), (int)((code >> 4) & 7u), ((code >> 7) & 31u) << 4);
    };

    v8i wf[8][2];
    {
        const uint8_t* gsrc;
        unsigned pitch;
        if (cross) {
            pitch = 64;
            gsrc = (const uint8_t*)par + (size_t)peer * REGION + R_W + g * (KD * 64);
        } else {
            if (AG_W_DELAY) __builtin_amdgcn_s_sleep(AG_W_DELAY);
            pitch = NSH;
            gsrc = pw + (c & 7) * 64;
        }
        gsrc += (size_t)(w * 512 + (lane >> 2)) * pitch + (lane & 3) * 16;
        const int grp = lane >> 4, i = lane & 15;
        const unsigned a0 = (unsigned)(((grp >> 1) * 32 + (i >> 1)) * 64 + (grp & 1) * 16 + (i & 1) * 8);
#pragma unroll
        for (int hf = 0; hf < 2; ++hf) {
            if (cross) AG_POLL(flags + (peer * 16 + g * 8) * 128, base + 16u * (hf + 1));
#pragma unroll
            for (int t = 0; t < 16; ++t)
                __builtin_amdgcn_global_load_lds((glb_void*)(gsrc + (size_t)((hf * 256 + t * 16) * pitch)), (lds_void*)(area + t * 1024), 16, 0, 16);
            wait_vm<0>();
#pragma unroll
            for (int s = 0; s < 4; ++s)
#pragma unroll
                for (int t = 0; t < 2; ++t)
#pragma unroll
                    for (int j = 0; j < 4; ++j) {
                        v2i r = __builtin_amdgcn_ds_read_tr8_b64_v2i32(
                            (__attribute__((address_space(3))) v2i*)(area + a0 + (s * 64 + j * 8) * 64 + t * 32));
                        wf[hf * 4 + s][t][2 * j] = r[0];
                        wf[hf * 4 + s][t][2 * j + 1] = r[1];
                    }
            wait_lgkm0();
            __builtin_amdgcn_sched_barrier(0);
        }
    }
    STAMP(1);
    __amdgpu_buffer_rsrc_t rx = mkrsrc(par, PAR_BYTES);
    const unsigned xo = (unsigned)w * 16384u + lane * 16u;
    const int m_w = lane & 31, h_w = lane >> 5;
    const int m_r = w * 4 + (lane >> 4), c_r = lane & 15;
    const unsigned rd_off = (unsigned)(m_r * 256 + ((c_r ^ (m_r & 7)) << 4));
    __amdgpu_buffer_rsrc_t ro = mkrsrc(cross ? p.out[q] : p.out[0], (unsigned)(NR * MSH * NSH * 4));
    const unsigned oo = (unsigned)((m_r * NSH + (cross ? (rank & 3) * 128 + g * 64 : (c & 7) * 64) + c_r * 4) * 4);
    u32x4 xf[16];

#define AG_MFMA(s, lo, hi) do { \
        v8i xa; \
        xa[0] = (lo)[0]; xa[1] = (lo)[1]; xa[2] = (lo)[2]; xa[3] = (lo)[3]; \
        xa[4] = (hi)[0]; xa[5] = (hi)[1]; xa[6] = (hi)[2]; xa[7] = (hi)[3]; \
        acc0 = __builtin_amdgcn_mfma_scale_f32_32x32x64_f8f6f4(wf[s][0], xa, acc0, 1, 1, 0, 0, 0, 0); \
        acc1 = __builtin_amdgcn_mfma_scale_f32_32x32x64_f8f6f4(wf[s][1], xa, acc1, 1, 1, 0, 0, 0, 0); } while (0)
    auto finish = [&](int par2, unsigned orow, f32x16& acc0, f32x16& acc1) {
        if (__builtin_expect(bad, 0)) {
            asm volatile("" ::: "memory");
#pragma unroll
            for (int i = 0; i < 16; ++i) acc0[i] = acc1[i] = __builtin_nanf("");
        }
        char* slot = area + par2 * 8192 + m_w * 256;
#pragma unroll
        for (int t = 0; t < 4; ++t) {
            f32x4 v0 = {acc0[4 * t], acc0[4 * t + 1], acc0[4 * t + 2], acc0[4 * t + 3]};
            f32x4 v1 = {acc1[4 * t], acc1[4 * t + 1], acc1[4 * t + 2], acc1[4 * t + 3]};
            *(f32x4*)(slot + (((2 * t + h_w) ^ (m_w & 7)) << 4)) = v0;
            *(f32x4*)(slot + 128 + (((2 * t + h_w) ^ (m_w & 7)) << 4)) = v1;
        }
        wait_lgkm0();
        __builtin_amdgcn_s_barrier();
        asm volatile("" ::: "memory");
        f32x4 part[8];
#pragma unroll
        for (int v = 0; v < 8; ++v) part[v] = *(const f32x4*)(lds + v * 16384 + par2 * 8192 + rd_off);
        const f32x4 sum = ((part[0] + part[1]) + (part[2] + part[3])) + ((part[4] + part[5]) + (part[6] + part[7]));
        f32x4 o;
#pragma unroll
        for (int i = 0; i < 4; ++i) {
            const float y = sum[i] * scale;
            const float yc = fminf(fmaxf(y, -60.0f), 60.0f);
            o[i] = y * __builtin_amdgcn_rcpf(1.0f + __expf(-yc));
        }
        __builtin_amdgcn_raw_buffer_store_b128(__builtin_bit_cast(u32x4, o), ro, oo + orow, 0, 17);
    };

    u32x4 rx4;
    rx4[0] = __builtin_amdgcn_readfirstlane((unsigned)(uintptr_t)par);
    rx4[1] = __builtin_amdgcn_readfirstlane((unsigned)((uintptr_t)par >> 32)) & 0xffffu;
    rx4[2] = PAR_BYTES;
    rx4[3] = 0x00020000u;
#pragma unroll
    for (int n = 0; n < 16; ++n) xf[n] = u32x4{0u, 0u, 0u, 0u};
    const unsigned vzero = 0;
    Step cur = next_step(), nxt = next_step();
    unsigned have = 0;
    unsigned fb = 0;
#define AG_IF "s_cmp_eq_u32 %[c], 0\n\ts_cbranch_scc1 .La%=\n\t"
#define AG_FI ".La%=:"
#define AG_FLOAD(cnd, f, st) asm volatile(AG_IF "global_load_dword %0, %1, %2 sc0 sc1\n" AG_FI \
        : "+v"(f) : "v"(vzero), "s"(flags + (st).foff), [c] "s"(cnd) : "scc", "memory")
#define AG_XLOAD4(cnd, n) asm volatile(AG_IF \
        "buffer_load_dwordx4 %0, %4, %5, %6 offen sc1\n\tbuffer_load_dwordx4 %1, %4, %5, %6 offen offset:1024 sc1\n\t" \
        "buffer_load_dwordx4 %2, %4, %5, %6 offen offset:2048 sc1\n\tbuffer_load_dwordx4 %3, %4, %5, %6 offen offset:3072 sc1\n" AG_FI \
        : "+v"(xf[n]), "+v"(xf[(n) + 1]), "+v"(xf[(n) + 2]), "+v"(xf[(n) + 3]) \
        : "v"(xo), "s"(rx4), "s"(cur.xoff + (n) * 1024u), [c] "s"(cnd) : "scc", "memory")
#define AG_LOAD_ALL(cnd) do { AG_XLOAD4(cnd, 0); AG_XLOAD4(cnd, 4); AG_FLOAD(cnd, fb, nxt); AG_XLOAD4(cnd, 8); AG_XLOAD4(cnd, 12); } while (0)
#define AG_STEP1(s, N) do { \
        asm volatile("s_waitcnt vmcnt(%2)" : "+v"(xf[2 * (s)]), "+v"(xf[2 * (s) + 1]) : "i"(N)); \
        AG_MFMA(s, xf[2 * (s)], xf[2 * (s) + 1]); \
        __builtin_amdgcn_sched_barrier(0); } while (0)
#define AG_STEP2(s, A, B) do { \
        asm volatile("s_cmp_lg_u32 %[go], 0\n\ts_cbranch_scc1 .Lw%=\n\ts_waitcnt vmcnt(%[b])\n.Lw%=:\n\ts_waitcnt vmcnt(%[a])" \
            : "+v"(xf[2 * (s)]), "+v"(xf[2 * (s) + 1]) : [go] "s"(go), [a] "i"(A), [b] "i"(B) : "scc"); \
        AG_MFMA(s, xf[2 * (s)], xf[2 * (s) + 1]); \
        __builtin_amdgcn_sched_barrier(0); } while (0)
#pragma unroll 1
    for (int i = 0; i < nstep; ++i) {
        if (!have) AG_POLL(flags + cur.foff, base + cur.need);
        {
            unsigned fresh = have ^ 1u;
            asm volatile("" : "+s"(fresh));
            AG_LOAD_ALL(fresh);
            asm volatile(AG_IF "global_store_dword %0, %0, %1 sc1\n" AG_FI :: "v"(vzero), "s"(ctr + 16 + w), [c] "s"(fresh) : "scc", "memory");
        }
        STAMP(2 + (i & 7) * 3);
        const unsigned orow = cur.orow;
        cur = nxt;
        nxt = next_step();
        f32x16 acc0 = {}, acc1 = {};
        AG_STEP1(0, 16); AG_STEP1(1, 14); AG_STEP1(2, 12); AG_STEP1(3, 10);
        asm volatile("s_waitcnt vmcnt(7)" : "+v"(fb));
        AG_STEP1(4, 7);
        const unsigned more = (unsigned)(i + 1 - nstep) >> 31;
        const unsigned go = (~(__builtin_amdgcn_readfirstlane(fb) - (base + cur.need)) >> 31) & more;
        unsigned fa = 0;
        AG_XLOAD4(go, 0);
        AG_FLOAD(go ^ 1u, fa, cur);
        AG_STEP2(5, 9, 6);
        AG_XLOAD4(go, 4);
        AG_FLOAD(go, fb, nxt);
        AG_STEP2(6, 12, 4);
        AG_XLOAD4(go, 8);
        AG_STEP2(7, 14, 2);
        AG_XLOAD4(go, 12);
        asm volatile("s_cmp_lg_u32 %[go], 0\n\ts_cbranch_scc1 .Lw%=\n\ts_waitcnt vmcnt(0)\n.Lw%=:" : "+v"(fa) : [go] "s"(go) : "scc");
        const unsigned second = (~(__builtin_amdgcn_readfirstlane(fa) - (base + cur.need)) >> 31) & more & ~go;
        AG_LOAD_ALL(second);
        have = go | second;
        __builtin_amdgcn_sched_barrier(0);
        finish(i & 1, orow, acc0, acc1);
        STAMP(4 + (i & 7) * 3);
    }
    wait_vm<0>();
#pragma unroll
    for (int n = 0; n < 16; ++n) asm volatile("" : "+v"(xf[n]));
#undef AG_STEP2
#undef AG_STEP1
#undef AG_LOAD_ALL
#undef AG_XLOAD4
#undef AG_FLOAD
#undef AG_IF
#undef AG_FI
    if (cross) {
        __builtin_amdgcn_s_barrier();
        if (tid == 0 && !bad)
            __hip_atomic_store((unsigned*)(wsp + DONE_OFF) + rank * 32 + c, call, __ATOMIC_RELAXED, __HIP_MEMORY_SCOPE_SYSTEM);
        STAMP(26);
    } else if (q != 0) {
        const Step st = make_step(peer, g, 7, 106u);
        const unsigned lbase = (unsigned)peer * REGION + R_LL + (unsigned)((g * 8 + w) * 8) * 1024u;
        const unsigned lo = lbase + lane * 16u, eo = lbase + 6144u + lane * 8u;
        const unsigned tagw = call << 24;
        u32x4 ll[6];
        u32x2 le;
        AG_POLL(flags + st.foff, base + st.need);
        STAMP(23);
#pragma unroll
        for (int n = 0; n < 10; ++n)
            xf[n] = __builtin_amdgcn_raw_buffer_load_b128(rx, xo + st.xoff + n * 1024, 0, 16);
        le = __builtin_amdgcn_raw_buffer_load_b64(rx, eo, 0, 17);
#pragma unroll
        for (int t = 0; t < 6; ++t) ll[t] = __builtin_amdgcn_raw_buffer_load_b128(rx, lo + t * 1024, 0, 17);
        __builtin_amdgcn_sched_barrier(0);
        f32x16 acc0 = {}, acc1 = {};
#pragma unroll
        for (int s = 0; s < 5; ++s) AG_MFMA(s, xf[2 * s], xf[2 * s + 1]);
        __builtin_amdgcn_sched_barrier(0);
        asm volatile("" : "+v"(acc0), "+v"(acc1));
#define AG_LL_STEP(s) do { \
            for (unsigned spin = 1;; ++spin) { \
                unsigned df = (ll[2 * (s)][3] ^ tagw) | (ll[2 * (s) + 1][3] ^ tagw); \
                if ((s) == 0) df |= le[1] ^ tagw; \
                if (__all((df >> 24) == 0u) || bad) break; \
                if ((spin & 63u) == 0u && (unsigned)__builtin_amdgcn_s_memrealtime() - t_start > DEADLINE_TICKS) bad = true; \
                if ((s) == 0) le = __builtin_amdgcn_raw_buffer_load_b64(rx, eo, 0, 17); \
                _Pragma("unroll") for (int t = 2 * (s); t < 6; ++t) ll[t] = __builtin_amdgcn_raw_buffer_load_b128(rx, lo + t * 1024, 0, 17); \
            } \
            u32x4 a = ll[2 * (s)], cc = ll[2 * (s) + 1]; \
            a[3] = (a[3] & 0x00ffffffu) | ((le[(s) >> 1] >> (16 * ((s) & 1))) << 24); \
            cc[3] = (cc[3] & 0x00ffffffu) | ((le[(s) >> 1] >> (16 * ((s) & 1) + 8)) << 24); \
            if ((s) == 2) STAMP(25); \
            AG_MFMA(5 + (s), a, cc); \
            asm volatile("" : "+v"(acc0), "+v"(acc1)); } while (0)
        AG_LL_STEP(0);
        AG_LL_STEP(1);
        AG_LL_STEP(2);
#undef AG_LL_STEP
        STAMP(24);
        finish(nstep & 1, st.orow, acc0, acc1);
        STAMP(26);
    } else {
        STAMP(26);
    }
#undef AG_POLL
#undef AG_MFMA
#undef AG_TAKEN
#undef AG_CLAIM_ROLE
    STAMP_FLUSH();
    return false;
}

__global__ void __launch_bounds__(NTHREADS, 2) ag_gemm_kernel(Params p) {
    __shared__ __attribute__((aligned(16))) char lds[LDS_BYTES];
    const int b = blockIdx.x;
    unsigned xcc;
    asm volatile("s_getreg_b32 %0, hwreg(HW_REG_XCC_ID)" : "=s"(xcc));
    const int q = (int)(xcc & 7u);
    const uint8_t* const px = p.x;
    const uint8_t* const pw = p.w;
    char* const self = p.ws[0];
    char* const wsp = p.ws[q];
    const int rank = p.rank;
    asm volatile("" :: "s"(px), "s"(pw), "s"(wsp), "s"(self), "s"(rank));
    const unsigned t_start = (unsigned)__builtin_amdgcn_s_memrealtime();
    if (__builtin_expect(!run_role<true>(p, lds, q, b >> 3, px, pw, self, wsp, rank, t_start, 0u AG_STAMP_PASS), 1)) return;

    lds_u32* sync_words = (lds_u32*)(lds + LDS_SYNC_OFF);
    const int lane = threadIdx.x & 63;
    wait_vm<0>();
    const unsigned cl = lds_load(sync_words + 8);
    if (threadIdx.x < 64) {
        bool ok = true;
        for (unsigned spin = 1; (int)(__builtin_amdgcn_readfirstlane(__hip_atomic_load((unsigned*)(self + ROLE_OFF), __ATOMIC_RELAXED, __HIP_MEMORY_SCOPE_AGENT)) - cl * 256u) < 0; ++spin) {
            if ((spin & 63u) == 0u && (unsigned)__builtin_amdgcn_s_memrealtime() - t_start > DEADLINE_TICKS) { ok = false; break; }
            __builtin_amdgcn_s_sleep(2);
        }
        unsigned k = 0, role = 0xffffffffu;
        if (lane == 0) {
            unsigned* tw = (unsigned*)(self + ROLE_OFF + 128);
            unsigned v = __hip_atomic_load(tw, __ATOMIC_RELAXED, __HIP_MEMORY_SCOPE_AGENT), n;
            do n = (v >> 8) == (cl & 0xffffffu) ? v + 1u : (cl << 8);
            while (!__hip_atomic_compare_exchange_strong(tw, &v, n, __ATOMIC_RELAXED, __ATOMIC_RELAXED, __HIP_MEMORY_SCOPE_AGENT));
            k = n & 255u;
        }
        k = __builtin_amdgcn_readfirstlane(k);
#pragma unroll 1
        for (int r0 = 0; r0 < 256 && role > 255u; r0 += 64) {
            const unsigned cw = __hip_atomic_load((unsigned*)(self + CTR_OFF) + (r0 + lane) * 32 + 8, __ATOMIC_RELAXED, __HIP_MEMORY_SCOPE_AGENT);
            unsigned long long m = __ballot(cw != cl);
            const unsigned cnt = (unsigned)__popcll(m);
            if (k < cnt) {
                for (; k; --k) m &= m - 1;
                role = (unsigned)(r0 + __ffsll((long long)m) - 1);
            } else {
                k -= cnt;
            }
        }
        if (lane == 0) lds_put_asm(sync_words + 10, ok ? role : 0xffffffffu);
        wait_lgkm0();
    }
    __builtin_amdgcn_s_barrier();
    asm volatile("" ::: "memory");
    const unsigned role = lds_load(sync_words + 10);
    if (role > 255u) return;
    run_role<false>(p, lds, (int)(role & 7u), (int)(role >> 3), px, pw, self, p.ws[role & 7u], rank, t_start, cl AG_STAMP_PASS);
}

}

extern "C" void kernel_launch(int rank, int nranks, ncclComm_t comm,
                              hipStream_t stream,
                              void* const* d_in, const int* in_sizes, int n_in,
                              void* d_out, int out_size,
                              const int* in_dtypes, int out_dtype,
                              void* const* d_out_all,
                              void* const* d_ws_all, size_t ws_bytes) {
    (void)comm; (void)in_dtypes; (void)out_dtype; (void)d_out;
    if (nranks != NR || (unsigned)rank >= (unsigned)NR || n_in < 4 || ws_bytes < WS_NEED ||
        in_sizes[0] != MSH * KD || in_sizes[1] != KD * NSH || out_size != NR * MSH * NSH) {
        fprintf(stderr, "kernel_launch: unexpected problem shape\n");
        return;
    }
    Params p;
    p.x = (const uint8_t*)d_in[0];
    p.w = (const uint8_t*)d_in[1];
    p.sx = (const float*)d_in[2];
    p.sw = (const float*)d_in[3];
    int vrank = 0;
    while (real_rank(vrank) != rank) ++vrank;
    for (int r = 0; r < NR; ++r) {
        p.ws[r] = (char*)d_ws_all[real_rank(vrank ^ r)];
        p.out[r] = (float*)d_out_all[real_rank(vrank ^ r)];
    }
    p.rank = vrank;
    hipLaunchKernelGGL(ag_gemm_kernel, dim3(NWG), dim3(NTHREADS), 0, stream, p);
    HIP_CHECK(hipGetLastError());
}
```
